```python
import numpy as np

kernel_name = "gpu_gemm_a2a_m4096_k4096_n4096_e4m3e4m3_dequant"


ABSMAX_RTOL = 0.03

COMPUTE_DTYPE = "float32"
EPILOGUE = "dequant"
IN_QUANT = None
OUT_QUANT = None
IN_FP8 = "e4m3e4m3"
IN_DTYPES = {"x": "e4m3", "w_mat": "e4m3",
             "scale_x": "float32", "scale_w": "float32"}
WORLD_SIZE = 8

def _snap_fp8(x, mant_bits, max_abs, min_normal):
    a = np.abs(x.astype(np.float64))
    _m, _e = np.frexp(a)
    q = np.where(a >= min_normal, np.ldexp(1.0, _e - 1 - mant_bits),
                 min_normal * 2.0 ** -mant_bits)
    snapped = np.minimum(np.round(a / q) * q, max_abs)
    return (np.sign(x) * snapped).astype(np.float32)

def setup_inputs():

    x = _snap_fp8(np.random.randn(4096, 4096).astype(np.float32), 3, 448.0, 2.0 ** -6)
    w_mat = _snap_fp8(np.random.randn(4096, 4096).astype(np.float32), 3, 448.0, 2.0 ** -6)
    scale_x = (0.02 * (1.0 + np.random.rand(1))).astype(np.float32)
    scale_w = (0.02 * (1.0 + np.random.rand(1))).astype(np.float32)
    return {"x": x, "w_mat": w_mat, "scale_x": scale_x, "scale_w": scale_w}

def reference(x, w_mat, scale_x, scale_w):
    acc = (x @ w_mat).astype(np.float32)
    y = (acc * (scale_x * scale_w)).astype(np.float32)
    return y
```

```cpp
#include <hip/hip_runtime.h>
#include <rccl/rccl.h>
#include <cstdio>

#define HIP_CHECK(expr) do { hipError_t _e = (expr); if (_e != hipSuccess) { \
    fprintf(stderr, "HIP error %s at %s:%d\n", hipGetErrorString(_e), __FILE__, __LINE__); \
    return; } } while (0)

#define XR 0
#define PUSH_PACE 2

namespace {

typedef int v2i __attribute__((ext_vector_type(2)));
typedef int v4i __attribute__((ext_vector_type(4)));
typedef int v8i __attribute__((ext_vector_type(8)));
typedef unsigned v4u __attribute__((ext_vector_type(4)));
typedef float v4f __attribute__((ext_vector_type(4)));
typedef __attribute__((address_space(3))) char lds_char;

constexpr int NR = 8;
constexpr int KD = 4096;
constexpr int ND = 4096;
constexpr int ML = 512;
constexpr int NSTEP = KD / 128;
constexpr int SLOTS = 4;
constexpr int PF = SLOTS - 1;
constexpr int WBYTES = 128 * 128;
constexpr int MT1 = 4;
constexpr int MT2 = 6;
constexpr int LDS_BYTES = SLOTS * (WBYTES + MT2 * 2048) + 64;
constexpr int NT_REM = (ML - XR) / 16;
constexpr int NT_DEL = 7 * XR / 16;
static_assert(XR % 16 == 0 && XR >= 0 && XR <= 96, "XR");
static_assert(NT_REM <= 8 * MT1 && NT_DEL <= 8 * MT2, "tiles");

constexpr size_t WS_CALL = 0;
constexpr size_t WS_XCNT = 2 * 256 * 256;
constexpr size_t WS_OCNT = WS_XCNT + 8 * 128;
constexpr size_t WS_TMO = WS_OCNT + 8 * 128;
constexpr size_t WS_DBG = 256 * 1024;
constexpr size_t WS_XS = 1u << 20;
constexpr size_t XS_BYTES = (size_t)7 * XR * KD;
constexpr size_t WS_NEED = WS_XS + 2 * XS_BYTES;
constexpr unsigned long long DEADLINE = 100000000ull;

struct Params {
  const unsigned char* x;
  const unsigned char* w;
  const float* sx;
  const float* sw;
  float* out[NR];
  unsigned char* ws[NR];
  int rank;
};

#define RSRC(p, n) __builtin_amdgcn_make_buffer_rsrc((void*)(p), (short)0, (int)(n), 0x00020000)
typedef __amdgpu_buffer_rsrc_t rsrc_t;

__device__ __forceinline__ unsigned long long now() { return __builtin_amdgcn_s_memrealtime(); }

template <int N> __device__ __forceinline__ void wait_vm_barrier() {
  asm volatile("s_waitcnt vmcnt(%0) lgkmcnt(0)\n\ts_barrier" ::"n"(N) : "memory");
}
__device__ __forceinline__ void wait_vm0() { asm volatile("s_waitcnt vmcnt(0)" ::: "memory"); }

__device__ __forceinline__ v4u srd(const void* p, unsigned bytes) {
  const unsigned long long a = (unsigned long long)p;
  return v4u{(unsigned)a, (unsigned)(a >> 32) & 0xffffu, bytes, 0x00020000u};
}
__device__ __forceinline__ void dma16(v4u r, unsigned dst, unsigned voff, unsigned soff) {
  asm volatile("s_mov_b32 m0, %0\n\ts_nop 0\n\tbuffer_load_dwordx4 %1, %2, %3 offen lds"
               ::"s"(dst), "v"(voff), "s"(r), "s"(soff) : "memory");
}

#define STAMP(idx) do {} while (0)

template <int MT>
__device__ __forceinline__ void gemm_tile(lds_char* lds, const unsigned char* xbase, unsigned xbytes, unsigned xrow0,
                                          int nt, const unsigned char* wbase, int wave, int lane, v4f (&acc)[MT][2]) {
  constexpr int SLOT_BYTES = WBYTES + MT * 2048;
  constexpr int XI = MT / 2;
  constexpr int CNT = 4 + XI;

  const unsigned rowoff = (unsigned)(lane >> 3) * 4096u;
  const unsigned c0 = (unsigned)((lane & 7) ^ ((lane >> 4) & 3)) * 16u;
  const unsigned v0 = rowoff + c0, v1 = rowoff + (c0 ^ 64u);
  const unsigned ldsb = (unsigned)(unsigned long long)lds;
  const int g = lane >> 4;
  const unsigned ax0 = (unsigned)((lane & 15) * 128 + ((g ^ ((lane >> 1) & 7)) * 16));
  const unsigned ax1 = ax0 ^ 64u;
  const int q = (lane & 15) >> 1;
  const unsigned aw0 = (unsigned)((16 * g + q) * 128 + (((2 * wave) ^ (((q >> 1) & 3) | ((g & 1) << 2))) * 16) + 8 * (lane & 1));
  const unsigned aw1 = aw0 ^ 16u;

  auto issue = [&](int k, int slot) {
    const bool live = k < NSTEP;
    const unsigned sb = ldsb + slot * SLOT_BYTES;
    const v4u wrk = srd(wbase, live ? (unsigned)(KD * ND - 3968) : 0u);
    const unsigned kw = (unsigned)(k * 128 + wave * 32) * 4096u;
#pragma unroll
    for (int e = 0; e < 4; ++e)
      dma16(wrk, sb + (wave * 4 + e) * 1024, (e & 2) ? v1 : v0, kw + e * 8 * 4096);
#pragma unroll
    for (int e = 0; e < XI; ++e) {
      const int idx = wave * XI + e;
      const v4u xr = srd(xbase, (live && (idx >> 1) < nt) ? xbytes : 0u);
      dma16(xr, sb + WBYTES + idx * 1024, (idx & 1) ? v1 : v0, xrow0 + (unsigned)idx * 8u * 4096u + (unsigned)k * 128u);
    }
  };
  auto rd = [&](int slot, v8i (&xf)[MT], v8i (&wf)[2]) {
    lds_char* sb = lds + slot * SLOT_BYTES;
#pragma unroll
    for (int u = 0; u < 2; ++u) {
      lds_char* a = sb + (u ? aw1 : aw0);
      v2i r0 = __builtin_amdgcn_ds_read_tr8_b64_v2i32((__attribute__((address_space(3))) v2i*)(a));
      v2i r1 = __builtin_amdgcn_ds_read_tr8_b64_v2i32((__attribute__((address_space(3))) v2i*)(a + 1024));
      v2i r2 = __builtin_amdgcn_ds_read_tr8_b64_v2i32((__attribute__((address_space(3))) v2i*)(a + 8192));
      v2i r3 = __builtin_amdgcn_ds_read_tr8_b64_v2i32((__attribute__((address_space(3))) v2i*)(a + 9216));
      wf[u] = v8i{r0[0], r0[1], r1[0], r1[1], r2[0], r2[1], r3[0], r3[1]};
    }
#pragma unroll
    for (int t = 0; t < MT; ++t) {
      v4i lo = *(__attribute__((address_space(3))) v4i*)(sb + WBYTES + t * 2048 + ax0);
      v4i hi = *(__attribute__((address_space(3))) v4i*)(sb + WBYTES + t * 2048 + ax1);
      xf[t] = v8i{lo[0], lo[1], lo[2], lo[3], hi[0], hi[1], hi[2], hi[3]};
    }
  };
  auto mma = [&](v8i (&xf)[MT], v8i (&wf)[2]) {
#pragma unroll
    for (int t = 0; t < MT; ++t)
#pragma unroll
      for (int u = 0; u < 2; ++u)
        acc[t][u] = __builtin_amdgcn_mfma_scale_f32_16x16x128_f8f6f4(wf[u], xf[t], acc[t][u], 0, 0, 0, 0, 0, 0);
  };

#pragma unroll
  for (int t = 0; t < MT; ++t) acc[t][0] = acc[t][1] = v4f{0.f, 0.f, 0.f, 0.f};
#pragma unroll
  for (int s = 0; s < PF; ++s) issue(s, s);

  v8i xa[MT], wa[2], xb[MT], wb[2];
#pragma unroll
  for (int t = 0; t < MT; ++t) xb[t] = v8i{0, 0, 0, 0, 0, 0, 0, 0};
  wb[0] = wb[1] = v8i{0, 0, 0, 0, 0, 0, 0, 0};
#pragma unroll 1
  for (int k = 0; k < NSTEP; k += 4) {
#define SB __builtin_amdgcn_sched_barrier(0)
    wait_vm_barrier<(PF - 1) * CNT>(); rd(0, xa, wa); SB; issue(k + 3, 3); SB; mma(xb, wb); SB;
    wait_vm_barrier<(PF - 1) * CNT>(); rd(1, xb, wb); SB; issue(k + 4, 0); SB; mma(xa, wa); SB;
    wait_vm_barrier<(PF - 1) * CNT>(); rd(2, xa, wa); SB; issue(k + 5, 1); SB; mma(xb, wb); SB;
    wait_vm_barrier<(PF - 1) * CNT>(); rd(3, xb, wb); SB; issue(k + 6, 2); SB; mma(xa, wa); SB;
  }
  mma(xb, wb);
}

template <int MT, class RowFn>
__device__ __forceinline__ void store_tile(rsrc_t dst, int nt, unsigned colbyte, float s, int lane, v4f (&acc)[MT][2], RowFn row0) {
  const unsigned voff = (unsigned)(lane & 7) * 2048u + colbyte + (unsigned)(lane & 8) * 8u + (unsigned)(lane >> 4) * 16u;
#pragma unroll
  for (int t = 0; t < MT; ++t) {
    if (t < nt) {
      v4u o1, o2;
#pragma unroll
      for (int r = 0; r < 4; ++r) {
        const int a = __builtin_bit_cast(int, acc[t][0][r] * s), b = __builtin_bit_cast(int, acc[t][1][r] * s);
        o1[r] = (unsigned)__builtin_amdgcn_update_dpp(a, b, 0x118, 0xf, 0xc, false);
        o2[r] = (unsigned)__builtin_amdgcn_update_dpp(b, a, 0x108, 0xf, 0x3, false);
      }
      const unsigned so = row0(t) * 2048u;
      __builtin_amdgcn_raw_buffer_store_b128(o1, dst, voff, so, 17);
      __builtin_amdgcn_raw_buffer_store_b128(o2, dst, voff, so + 8u * 2048u, 17);
    }
  }
}

__device__ __forceinline__ bool wait_counters(const unsigned char* myws, size_t base, bool need, unsigned target,
                                              unsigned call, int lane) {
  unsigned* tmo = (unsigned*)(myws + WS_TMO);
  const unsigned* c = (const unsigned*)(myws + base + (size_t)(lane & 7) * 128);
  const unsigned long long t0 = now();
  for (;;) {
    const unsigned v = __hip_atomic_load(c, __ATOMIC_RELAXED, __HIP_MEMORY_SCOPE_SYSTEM);
    if (__all(!need || (int)(v - target) >= 0)) return true;
    if (__hip_atomic_load(tmo, __ATOMIC_RELAXED, __HIP_MEMORY_SCOPE_AGENT) == call || now() - t0 > DEADLINE) {
      if (lane == 0) __hip_atomic_store(tmo, call, __ATOMIC_RELAXED, __HIP_MEMORY_SCOPE_AGENT);
      return false;
    }
    __builtin_amdgcn_s_sleep(1);
  }
}

}

__global__ void __launch_bounds__(320, 1) gemm_a2a_kernel(Params P) {
  extern __shared__ __attribute__((aligned(16))) char smem[];
  lds_char* lds = (lds_char*)smem;
  const int lane = threadIdx.x & 63;
  const int wave = __builtin_amdgcn_readfirstlane(threadIdx.x >> 6);
  const int b = blockIdx.x, lab = b & 7, idx = b >> 3, ct = idx & 3, rg = idx >> 2;
  const int rank = P.rank & 7;
  unsigned char* myws = P.ws[0];
  unsigned char* dws = P.ws[lab];
  float* dout = P.out[lab];
  const unsigned char* px = P.x;
  const unsigned char* pw = P.w;
  const float* psx = P.sx;
  const float* psw = P.sw;
  asm volatile("" ::"s"(rank), "s"(myws), "s"(dws), "s"(dout), "s"(px), "s"(pw), "s"(psx), "s"(psw));
  const int dest = (rank + lab) & 7;

  if (wave == 4) {
    if (XR == 0 || lab == 0) return;
    constexpr int NP = XR / 8;
    unsigned call = __hip_atomic_fetch_add((unsigned*)(myws + WS_CALL + (size_t)b * 256) + lane, 1u, __ATOMIC_RELAXED, __HIP_MEMORY_SCOPE_AGENT);
    const rsrc_t xs = RSRC(px + (size_t)(ML - XR) * KD, XR * KD);
    const unsigned off = (unsigned)idx * (NP * 1024u) + (unsigned)lane * 16u;
    v4u d[NP > 0 ? NP : 1];
#pragma unroll
    for (int i = 0; i < NP; ++i) d[i] = __builtin_amdgcn_raw_buffer_load_b128(xs, off, i * 1024, 0);
    call = __builtin_amdgcn_readfirstlane(call) + 1u;
    const rsrc_t xd = RSRC(dws + WS_XS + (size_t)(call & 1u) * XS_BYTES + (size_t)(7 - lab) * XR * KD, XR * KD);
    if (PUSH_PACE == 0) {
#pragma unroll
      for (int i = 0; i < NP; ++i) __builtin_amdgcn_raw_buffer_store_b128(d[i], xd, off, i * 1024, 17);
#pragma unroll 1
      for (int k = 0; k < NSTEP; ++k) __builtin_amdgcn_s_barrier();
    } else {
#pragma unroll
      for (int k = 0; k < NSTEP; ++k) {
        if (k % (PUSH_PACE > 0 ? PUSH_PACE : 1) == 0 && k / (PUSH_PACE > 0 ? PUSH_PACE : 1) < NP)
          __builtin_amdgcn_raw_buffer_store_b128(d[k / (PUSH_PACE > 0 ? PUSH_PACE : 1)], xd, off, (k / (PUSH_PACE > 0 ? PUSH_PACE : 1)) * 1024, 17);
        asm volatile("s_barrier" ::: "memory");
      }
    }
    wait_vm0();
    STAMP(4);
    if (lane == 0) __hip_atomic_fetch_add((unsigned*)(dws + WS_XCNT + (size_t)rank * 128), 1u, __ATOMIC_RELAXED, __HIP_MEMORY_SCOPE_SYSTEM);
    return;
  }

  unsigned call = 0;
  if (lab == 0 && wave == 0)
    call = __hip_atomic_fetch_add((unsigned*)(myws + WS_CALL + 65536 + (size_t)b * 256) + lane, 1u, __ATOMIC_RELAXED, __HIP_MEMORY_SCOPE_AGENT);
  const float scale = psx[0] * psw[0];
  const unsigned char* wpanel = pw + dest * ML + ct * 128;
  const unsigned colbyte = (unsigned)(ct * 128 + wave * 32) * 4u;
  {
    const int t0 = lab == 0 ? rg * 4 : (NT_REM * rg) >> 3;
    const int nt = lab == 0 ? 4 : ((NT_REM * (rg + 1)) >> 3) - t0;
    v4f acc[MT1][2];
    gemm_tile<MT1>(lds, px, ML * KD, (unsigned)t0 * 16u * KD, nt, wpanel, wave, lane, acc);
    call = __builtin_amdgcn_readfirstlane(call) + 1u;
    if (wave == 0) STAMP(1);
    const rsrc_t dst = RSRC(dout + (size_t)rank * ML * ML, ML * ML * 4);
    store_tile<MT1>(dst, nt, colbyte, scale, lane, acc, [&](int t) { return (unsigned)(t0 + t) * 16u; });
  }
  if (lab != 0) {
    wait_vm0();
    if (wave == 0) STAMP(2);
    if (lane == 0) __hip_atomic_fetch_add((unsigned*)(dws + WS_OCNT + (size_t)rank * 128), 1u, __ATOMIC_RELAXED, __HIP_MEMORY_SCOPE_SYSTEM);
    return;
  }

  if (XR > 0) {
    const int t0 = (NT_DEL * rg) >> 3;
    const int nt = ((NT_DEL * (rg + 1)) >> 3) - t0;
    __attribute__((address_space(3))) int* flag = (__attribute__((address_space(3))) int*)(lds + LDS_BYTES - 64);
    if (wave == 0) {
      constexpr int TPS = XR > 0 ? XR / 16 : 1;
      const int slot = (lane - rank - 1) & 7;
      const bool need = lane < 8 && slot < 7 && slot >= t0 / TPS && slot <= (t0 + nt - 1) / TPS;
      const bool ok = wait_counters(myws, WS_XCNT, need, call * 32u, call, lane);
      __builtin_amdgcn_fence(__ATOMIC_ACQUIRE, "");
      if (lane == 0) { flag[0] = ok ? 1 : 0; flag[1] = (int)call; }
    }
    asm volatile("s_waitcnt vmcnt(0) lgkmcnt(0)\n\ts_barrier" ::: "memory");
    const bool ok = __builtin_amdgcn_readfirstlane(flag[0]) != 0;
    call = (unsigned)__builtin_amdgcn_readfirstlane(flag[1]);
    if (wave == 0) STAMP(3);
    v4f acc[MT2][2];
    gemm_tile<MT2>(lds, myws + WS_XS + (size_t)(call & 1u) * XS_BYTES, ok ? (unsigned)XS_BYTES : 0u, (unsigned)t0 * 16u * KD, nt, wpanel, wave, lane, acc);
    const rsrc_t dst = RSRC(dout, NR * ML * ML * 4);
    store_tile<MT2>(dst, nt, colbyte, ok ? scale : __builtin_nanf(""), lane, acc, [&](int t) {
      constexpr int TPS = XR > 0 ? XR / 16 : 1;
      const int tt = t0 + t, slot = tt / TPS;
      return (unsigned)(((rank + 1 + slot) & 7) * ML + (ML - XR) + (tt - slot * TPS) * 16);
    });
    if (wave == 0) STAMP(2);
  }
  if (b == 0 && wave == 0) {
    wait_vm0();
    const bool ok = wait_counters(myws, WS_OCNT, lane < 8 && lane != rank, call * 128u, call, lane);
    if (!ok && lane == 0) dout[(size_t)rank * ML * ML] = __builtin_nanf("");
    STAMP(5);
  }
}

extern "C" void kernel_launch(int rank, int nranks, ncclComm_t comm, hipStream_t stream,
                              void* const* d_in, const int* in_sizes, int n_in,
                              void* d_out, int out_size,
                              const int* in_dtypes, int out_dtype,
                              void* const* d_out_all, void* const* d_ws_all, size_t ws_bytes) {
  (void)comm; (void)in_dtypes; (void)out_dtype; (void)d_out;
  if (nranks != NR || n_in != 4 || in_sizes[0] != ML * KD || in_sizes[1] != KD * ND || out_size != NR * ML * ML ||
      ws_bytes < WS_NEED || (unsigned)rank >= (unsigned)NR) {
    fprintf(stderr, "kernel_launch: unexpected problem shape\n");
    return;
  }
  static bool init = false;
  if (!init) {
    HIP_CHECK(hipFuncSetAttribute((const void*)gemm_a2a_kernel, hipFuncAttributeMaxDynamicSharedMemorySize, LDS_BYTES));
    init = true;
  }
  Params P;
  P.x = (const unsigned char*)d_in[0];
  P.w = (const unsigned char*)d_in[1];
  P.sx = (const float*)d_in[2];
  P.sw = (const float*)d_in[3];
  for (int p = 0; p < NR; ++p) {
    P.out[p] = (float*)d_out_all[(rank + p) % NR];
    P.ws[p] = (unsigned char*)d_ws_all[(rank + p) % NR];
  }
  P.rank = rank;
  hipLaunchKernelGGL(gemm_a2a_kernel, dim3(256), dim3(320), LDS_BYTES, stream, P);
  HIP_CHECK(hipGetLastError());
}
```

```cpp
#include <hip/hip_runtime.h>
#include <rccl/rccl.h>
#include <cstdio>

#define HIP_CHECK(expr) do { hipError_t _e = (expr); if (_e != hipSuccess) { \
    fprintf(stderr, "HIP error %s at %s:%d\n", hipGetErrorString(_e), __FILE__, __LINE__); \
    return; } } while (0)

#define XR 96
#define OWIN 1
#define PWIN 4
#define ROT_L 4
#define ROT_RG 0
#define ROT_CT 0
#define KMUL 1
#define RG_SH 0
#define SLOTS 4

namespace {

typedef int v2i __attribute__((ext_vector_type(2)));
typedef int v4i __attribute__((ext_vector_type(4)));
typedef int v8i __attribute__((ext_vector_type(8)));
typedef unsigned v4u __attribute__((ext_vector_type(4)));
typedef float v4f __attribute__((ext_vector_type(4)));
typedef __attribute__((address_space(3))) char lds_char;

constexpr int NR = 8;
constexpr int KD = 4096;
constexpr int ND = 4096;
constexpr int ML = 512;
constexpr int NSTEP = KD / 128;
constexpr int WBYTES = 128 * 128;
constexpr int MT = 4;
constexpr int SLOT_BYTES = WBYTES + MT * 2048;
constexpr int LDS_BYTES = SLOTS * SLOT_BYTES + 64;
constexpr int NT_REM = (ML - XR) / 16;
constexpr int TPS = XR / 16;
constexpr int TPS_A = (TPS + 1) / 2;
static_assert(XR % 16 == 0 && XR >= 64 && XR <= 128, "XR");
static_assert(NT_REM <= 7 * MT && TPS_A <= MT, "tiles");
static_assert(LDS_BYTES <= 160 * 1024 && LDS_BYTES > 80 * 1024, "one workgroup per CU");

constexpr size_t WS_CALL = 0;
constexpr size_t WS_XCNT = 2048 * 128;
constexpr size_t WS_OCNT = WS_XCNT + 8 * 128;
constexpr size_t WS_TMO = WS_OCNT + 8 * 128;
constexpr size_t WS_DBG = 512 * 1024;
constexpr size_t WS_XS = 1u << 20;
constexpr size_t XS_BYTES = (size_t)7 * XR * KD;
constexpr size_t WS_NEED = WS_XS + 2 * XS_BYTES;
constexpr unsigned long long DEADLINE = 100000000ull;

struct Params {
  const unsigned char* x;
  const unsigned char* w;
  const float* sx;
  const float* sw;
  float* out[NR];
  unsigned char* ws[NR];
  int rank;
};

#define RSRC(p, n) __builtin_amdgcn_make_buffer_rsrc((void*)(p), (short)0, (int)(n), 0x00020000)
typedef __amdgpu_buffer_rsrc_t rsrc_t;

__device__ __forceinline__ unsigned long long now() { return __builtin_amdgcn_s_memrealtime(); }

template <int N> __device__ __forceinline__ void wait_vm_barrier() {
  asm volatile("s_waitcnt vmcnt(%0) lgkmcnt(0)\n\ts_barrier" ::"n"(N) : "memory");
}
__device__ __forceinline__ void wait_lgkm_barrier() { asm volatile("s_waitcnt lgkmcnt(0)\n\ts_barrier" ::: "memory"); }
template <int N> __device__ __forceinline__ void wait_vm() { asm volatile("s_waitcnt vmcnt(%0)" ::"n"(N) : "memory"); }
__device__ __forceinline__ void wait_vm0() { asm volatile("s_waitcnt vmcnt(0)" ::: "memory"); }

__device__ __forceinline__ v4u srd(const void* p, unsigned bytes) {
  const unsigned long long a = (unsigned long long)p;
  return v4u{(unsigned)a, (unsigned)(a >> 32) & 0xffffu, bytes, 0x00020000u};
}
__device__ __forceinline__ void dma16(v4u r, unsigned dst, unsigned voff, unsigned soff) {
  asm volatile("s_mov_b32 m0, %0\n\ts_nop 0\n\tbuffer_load_dwordx4 %1, %2, %3 offen lds"
               ::"s"(dst), "v"(voff), "s"(r), "s"(soff) : "memory");
}

#define STAMP(idx) do {} while (0)

__device__ __forceinline__ void load_tile(lds_char* lds, const unsigned char* xbase, unsigned xbytes, unsigned xrow0,
                                          int nt, const unsigned char* wbase, int krot, int wave, int lane) {
  constexpr int PF = SLOTS - 1;
  constexpr int XI = MT / 2;
  constexpr int CNT = 4 + XI;
  static_assert((PF - 1) * CNT <= 63, "vmcnt");
  const unsigned rowoff = (unsigned)(lane >> 3) * 4096u;
  const unsigned c0 = (unsigned)((lane & 7) ^ ((lane >> 4) & 3)) * 16u;
  const unsigned v0 = rowoff + c0, v1 = rowoff + (c0 ^ 64u);
  const unsigned ldsb = (unsigned)(unsigned long long)lds;
  const v4u wr = srd(wbase, (unsigned)(KD * ND - 3968));
  v4u xr[XI];
  unsigned xv[XI];
#pragma unroll
  for (int e = 0; e < XI; ++e) {
    const int idx = wave * XI + e;
    xr[e] = srd(xbase, (idx >> 1) < nt ? xbytes : 0u);
    xv[e] = ((idx & 1) ? v1 : v0) + (unsigned)idx * 8u * 4096u;
  }
  auto issue = [&](int k, unsigned so) {
    const unsigned kk = ((unsigned)(k + krot) * KMUL) & (NSTEP - 1);
    const unsigned sb = ldsb + so;
    const unsigned kw = (kk * 128u + (unsigned)wave * 32u) * 4096u;
#pragma unroll
    for (int e = 0; e < 4; ++e)
      dma16(wr, sb + (wave * 4 + e) * 1024, (e & 2) ? v1 : v0, kw + e * 8 * 4096);
#pragma unroll
    for (int e = 0; e < XI; ++e)
      dma16(xr[e], sb + WBYTES + (wave * XI + e) * 1024, xv[e], xrow0 + kk * 128u);
  };
#pragma unroll
  for (int s = 0; s < PF; ++s) issue(s, s * SLOT_BYTES);
  unsigned prev = PF * SLOT_BYTES;
#pragma unroll 1
  for (int k = 0; k < NSTEP - PF; ++k) {
    wait_vm_barrier<(PF - 1) * CNT>();
    issue(k + PF, prev);
    prev = (prev + SLOT_BYTES == SLOTS * SLOT_BYTES) ? 0u : prev + SLOT_BYTES;
  }
  if (PF > 1) wait_vm_barrier<(PF > 1 ? PF - 1 : 0) * CNT>();
  if (PF > 2) wait_vm_barrier<(PF > 2 ? PF - 2 : 0) * CNT>();
  if (PF > 3) wait_vm_barrier<(PF > 3 ? PF - 3 : 0) * CNT>();
  if (PF > 4) wait_vm_barrier<(PF > 4 ? PF - 4 : 0) * CNT>();
  static_assert(PF <= 5, "tail");
  wait_vm_barrier<0>();
}

__device__ __forceinline__ void mma_tile(lds_char* lds, int wave, int lane, v4f (&acc)[MT][2]) {
  const int g = lane >> 4;
  const unsigned ax0 = (unsigned)((lane & 15) * 128 + ((g ^ ((lane >> 1) & 7)) * 16)) + WBYTES;
  const unsigned ax1 = ax0 ^ 64u;
  const int q = (lane & 15) >> 1;
  const unsigned aw0 = (unsigned)((16 * g + q) * 128 + (((2 * wave) ^ (((q >> 1) & 3) | ((g & 1) << 2))) * 16) + 8 * (lane & 1));
  const unsigned aw1 = aw0 ^ 16u;
  auto rd = [&](unsigned so, v8i (&xf)[MT], v8i (&wf)[2]) {
#pragma unroll
    for (int u = 0; u < 2; ++u) {
      lds_char* a = lds + ((u ? aw1 : aw0) + so);
      v2i r0 = __builtin_amdgcn_ds_read_tr8_b64_v2i32((__attribute__((address_space(3))) v2i*)(a));
      v2i r1 = __builtin_amdgcn_ds_read_tr8_b64_v2i32((__attribute__((address_space(3))) v2i*)(a + 1024));
      v2i r2 = __builtin_amdgcn_ds_read_tr8_b64_v2i32((__attribute__((address_space(3))) v2i*)(a + 8192));
      v2i r3 = __builtin_amdgcn_ds_read_tr8_b64_v2i32((__attribute__((address_space(3))) v2i*)(a + 9216));
      wf[u] = v8i{r0[0], r0[1], r1[0], r1[1], r2[0], r2[1], r3[0], r3[1]};
    }
    lds_char* a0 = lds + (ax0 + so);
    lds_char* a1 = lds + (ax1 + so);
#pragma unroll
    for (int t = 0; t < MT; ++t) {
      v4i lo = *(__attribute__((address_space(3))) v4i*)(a0 + t * 2048);
      v4i hi = *(__attribute__((address_space(3))) v4i*)(a1 + t * 2048);
      xf[t] = v8i{lo[0], lo[1], lo[2], lo[3], hi[0], hi[1], hi[2], hi[3]};
    }
  };
  auto mma = [&](v8i (&xf)[MT], v8i (&wf)[2]) {
#pragma unroll
    for (int t = 0; t < MT; ++t)
#pragma unroll
      for (int u = 0; u < 2; ++u)
        acc[t][u] = __builtin_amdgcn_mfma_scale_f32_16x16x128_f8f6f4(wf[u], xf[t], acc[t][u], 0, 0, 0, 0, 0, 0);
  };
#pragma unroll
  for (int t = 0; t < MT; ++t) acc[t][0] = acc[t][1] = v4f{0.f, 0.f, 0.f, 0.f};
  v8i xa[MT], wa[2], xb[MT], wb[2];
#pragma unroll
  for (int t = 0; t < MT; ++t) xb[t] = v8i{0, 0, 0, 0, 0, 0, 0, 0};
  wb[0] = wb[1] = v8i{0, 0, 0, 0, 0, 0, 0, 0};
  unsigned cur = 0;
#define SB __builtin_amdgcn_sched_barrier(0)
#define NEXT cur = (cur + SLOT_BYTES == SLOTS * SLOT_BYTES) ? 0u : cur + SLOT_BYTES
#pragma unroll 1
  for (int k = 0; k < NSTEP; k += 2) {
    wait_lgkm_barrier(); rd(cur, xa, wa); SB; mma(xb, wb); SB; NEXT;
    wait_lgkm_barrier(); rd(cur, xb, wb); SB; mma(xa, wa); SB; NEXT;
  }
  mma(xb, wb);
}

__device__ __forceinline__ void store_tile(rsrc_t dst, int nt, unsigned so, bool remote, float s, int lane, v4f (&acc)[MT][2]) {
  const unsigned voff = (unsigned)(lane & 7) * 2048u + (unsigned)(lane & 8) * 8u + (unsigned)(lane >> 4) * 16u;
#pragma unroll
  for (int t = 0; t < MT; ++t) {
    if (t < nt) {
      v4u o1, o2;
#pragma unroll
      for (int r = 0; r < 4; ++r) {
        const int a = __builtin_bit_cast(int, acc[t][0][r] * s), b = __builtin_bit_cast(int, acc[t][1][r] * s);
        o1[r] = (unsigned)__builtin_amdgcn_update_dpp(a, b, 0x118, 0xf, 0xc, false);
        o2[r] = (unsigned)__builtin_amdgcn_update_dpp(b, a, 0x108, 0xf, 0x3, false);
      }
      if (remote) wait_vm<OWIN - 1>();
      __builtin_amdgcn_raw_buffer_store_b128(o1, dst, voff, so + (unsigned)t * 16u * 2048u, 17);
      if (remote) wait_vm<OWIN - 1>();
      __builtin_amdgcn_raw_buffer_store_b128(o2, dst, voff, so + (unsigned)t * 16u * 2048u + 8u * 2048u, 17);
    }
  }
}

__device__ __forceinline__ bool wait_counters(const unsigned char* myws, size_t base, bool need, unsigned target,
                                              unsigned call, int lane) {
  unsigned* tmo = (unsigned*)(myws + WS_TMO);
  const unsigned* c = (const unsigned*)(myws + base + (size_t)(lane & 7) * 128);
  const unsigned long long t0 = now();
  for (;;) {
    const unsigned v = __hip_atomic_load(c, __ATOMIC_RELAXED, __HIP_MEMORY_SCOPE_SYSTEM);
    if (__all(!need || (int)(v - target) >= 0)) return true;
    if (__hip_atomic_load(tmo, __ATOMIC_RELAXED, __HIP_MEMORY_SCOPE_AGENT) == call || now() - t0 > DEADLINE) {
      if (lane == 0) __hip_atomic_store(tmo, call, __ATOMIC_RELAXED, __HIP_MEMORY_SCOPE_AGENT);
      return false;
    }
    __builtin_amdgcn_s_sleep(1);
  }
}

}

__global__ void __launch_bounds__(512, 2) gemm_a2a_kernel(Params P) {
  extern __shared__ __attribute__((aligned(16))) char smem[];
  lds_char* lds = (lds_char*)smem;
  const int lane = threadIdx.x & 63;
  const int wave8 = __builtin_amdgcn_readfirstlane(threadIdx.x >> 6);
  const int wave = wave8 & 3;
  const int b = blockIdx.x, lab = b & 7, idx = b >> 3, ct = idx & 3, rg = idx >> 2;
  const int rank = P.rank & 7;
  unsigned char* myws = P.ws[0];
  unsigned char* dws = P.ws[lab];
  float* myout = P.out[0];
  float* dout = P.out[lab];
  const unsigned char* px = P.x;
  const unsigned char* pw = P.w;
  const float* psx = P.sx;
  const float* psw = P.sw;
  asm volatile("" ::"s"(rank), "s"(myws), "s"(dws), "s"(myout), "s"(dout), "s"(px), "s"(pw), "s"(psx), "s"(psw));
  const bool pusher = lab != 0 && rg == 7;
  const bool has2 = pusher || (lab == 0 && rg < 7 && TPS > TPS_A);
  const int slot2 = pusher ? lab - 1 : rg;
  __attribute__((address_space(3))) int* flag = (__attribute__((address_space(3))) int*)(lds + LDS_BYTES - 64);

  unsigned call = 0;
  if (pusher || (lab == 0 && wave8 == 0))
    call = __hip_atomic_fetch_add((unsigned*)(myws + WS_CALL + (size_t)(b * 8 + wave8) * 128) + (lane & 31), 1u, __ATOMIC_RELAXED, __HIP_MEMORY_SCOPE_AGENT);
  float sxv = 0.f, swv = 0.f;
  if (wave8 < 4) { sxv = psx[0]; swv = psw[0]; }

  if (pusher) {
    constexpr int NP = XR / 8;
    const rsrc_t xs = RSRC(px + (size_t)(ML - XR) * KD, XR * KD);
    const unsigned off = (unsigned)(ct * 8 + wave8) * 1024u + (unsigned)lane * 16u;
    v4u d[NP];
#pragma unroll
    for (int i = 0; i < NP; ++i) d[i] = __builtin_amdgcn_raw_buffer_load_b128(xs, off, i * 32768, 0);
    call = (__builtin_amdgcn_readfirstlane(call) >> 1) + 1u;
    const rsrc_t xd = RSRC(dws + WS_XS + (size_t)(call & 1u) * XS_BYTES + (size_t)(7 - lab) * XR * KD, XR * KD);
#pragma unroll
    for (int i = 0; i < NP; ++i) {
      if (i >= PWIN) wait_vm<PWIN - 1>();
      __builtin_amdgcn_raw_buffer_store_b128(d[i], xd, off, i * 32768, 17);
    }
    wait_vm0();
    if (wave8 == 0) STAMP(4);
    if (lane == 0) __hip_atomic_fetch_add((unsigned*)(dws + WS_XCNT + (size_t)rank * 128), 1u, __ATOMIC_RELAXED, __HIP_MEMORY_SCOPE_SYSTEM);
  } else {
    call = (__builtin_amdgcn_readfirstlane(call) >> 1) + 1u;
  }
  asm volatile("" : "+v"(sxv), "+v"(swv));
  float scale = sxv * swv;

#pragma unroll 1
  for (int job = pusher ? 1 : 0; job < (has2 ? 2 : 1); ++job) {
    const unsigned char* xbase;
    unsigned xbytes, xrow0, orow;
    int nt, wcol;
    float* obase;
    if (job == 0) {
      const int t1 = lab == 0 ? rg * 4 : (NT_REM * rg) / 7;
      nt = lab == 0 ? 4 : (NT_REM * (rg + 1)) / 7 - t1;
      xbase = px; xbytes = ML * KD; xrow0 = (unsigned)t1 * 16u;
      wcol = ((rank + lab) & 7) * ML;
      obase = dout; orow = (unsigned)rank * ML + xrow0;
    } else {
      const int src = (rank + 1 + slot2) & 7;
      if (wave8 == 0) {
        const bool ok = wait_counters(myws, WS_XCNT, lane == src, call * 32u, call, lane);
        __builtin_amdgcn_fence(__ATOMIC_ACQUIRE, "");
        wait_vm0();
        if (lane == 0) { flag[0] = ok ? 1 : 0; flag[1] = (int)call; }
      }
      wait_lgkm_barrier();
      const bool ok = __builtin_amdgcn_readfirstlane(flag[0]) != 0;
      call = (unsigned)__builtin_amdgcn_readfirstlane(flag[1]);
      if (wave8 == 0) STAMP(3);
      const int tofs = pusher ? 0 : TPS_A;
      nt = pusher ? TPS_A : TPS - TPS_A;
      xbase = myws + WS_XS + (size_t)(call & 1u) * XS_BYTES; xbytes = ok ? (unsigned)XS_BYTES : 0u;
      xrow0 = (unsigned)(slot2 * XR + tofs * 16);
      wcol = rank * ML;
      obase = myout; orow = (unsigned)(src * ML + (ML - XR) + tofs * 16);
      if (!ok) scale = __builtin_nanf("");
    }
    if (wave8 >= 4) {
      load_tile(lds, xbase, xbytes, xrow0 * KD, nt, pw + wcol + ct * 128, lab * ROT_L + (rg >> RG_SH) * ROT_RG + ct * ROT_CT, wave, lane);
    } else {
      v4f acc[MT][2];
      mma_tile(lds, wave, lane, acc);
      store_tile(RSRC(obase, NR * ML * ML * 4), nt, orow * 2048u + (unsigned)(ct * 128 + wave * 32) * 4u, job == 0 && lab != 0, scale, lane, acc);
    }
  }
  if (wave8 >= 4) return;
  if (lab != 0 && !pusher) {
    wait_vm0();
    if (wave == 0) STAMP(2);
    if (lane == 0) __hip_atomic_fetch_add((unsigned*)(dws + WS_OCNT + (size_t)rank * 128), 1u, __ATOMIC_RELAXED, __HIP_MEMORY_SCOPE_SYSTEM);
    return;
  }
  if (wave == 0 && has2) STAMP(2);
  if (b == 28 * 8 && wave == 0) {
    wait_vm0();
    const bool ok = wait_counters(myws, WS_OCNT, lane < 8 && lane != rank, call * 112u, call, lane);
    if (!ok && lane == 0) myout[(size_t)(rank * ML + 448) * ML] = __builtin_nanf("");
    STAMP(5);
  }
}

extern "C" void kernel_launch(int rank, int nranks, ncclComm_t comm, hipStream_t stream,
                              void* const* d_in, const int* in_sizes, int n_in,
                              void* d_out, int out_size,
                              const int* in_dtypes, int out_dtype,
                              void* const* d_out_all, void* const* d_ws_all, size_t ws_bytes) {
  (void)comm; (void)in_dtypes; (void)out_dtype; (void)d_out;
  if (nranks != NR || n_in != 4 || in_sizes[0] != ML * KD || in_sizes[1] != KD * ND || out_size != NR * ML * ML ||
      ws_bytes < WS_NEED || (unsigned)rank >= (unsigned)NR) {
    fprintf(stderr, "kernel_launch: unexpected problem shape\n");
    return;
  }
  static bool init = false;
  if (!init) {
    HIP_CHECK(hipFuncSetAttribute((const void*)gemm_a2a_kernel, hipFuncAttributeMaxDynamicSharedMemorySize, LDS_BYTES));
    init = true;
  }
  Params P;
  P.x = (const unsigned char*)d_in[0];
  P.w = (const unsigned char*)d_in[1];
  P.sx = (const float*)d_in[2];
  P.sw = (const float*)d_in[3];
  for (int p = 0; p < NR; ++p) {
    P.out[p] = (float*)d_out_all[(rank + p) % NR];
    P.ws[p] = (unsigned char*)d_ws_all[(rank + p) % NR];
  }
  P.rank = rank;
  hipLaunchKernelGGL(gemm_a2a_kernel, dim3(256), dim3(512), LDS_BYTES, stream, P);
  HIP_CHECK(hipGetLastError());
}
```

```cpp
#include <hip/hip_runtime.h>
#include <rccl/rccl.h>
#include <cstdio>

#define HIP_CHECK(expr) do { hipError_t _e = (expr); if (_e != hipSuccess) { \
    fprintf(stderr, "HIP error %s at %s:%d\n", hipGetErrorString(_e), __FILE__, __LINE__); \
    return; } } while (0)

#define XR 128
#define NRG 6
#define OWIN 1
#define PWIN 1
#define TAILW 2
#define TAILN 2
#define ROT_L 4
#define ROT_RG 1
#define ROT_CT 0
#define ROT2_L 2
#define ROT2_CT 0
#define SLOTS 4
#define ILV 1

namespace {

typedef int v2i __attribute__((ext_vector_type(2)));
typedef int v4i __attribute__((ext_vector_type(4)));
typedef int v8i __attribute__((ext_vector_type(8)));
typedef unsigned v4u __attribute__((ext_vector_type(4)));
typedef float v4f __attribute__((ext_vector_type(4)));
typedef __attribute__((address_space(3))) char lds_char;

constexpr int NR = 8;
constexpr int KD = 4096;
constexpr int ND = 4096;
constexpr int ML = 512;
constexpr int NSTEP = KD / 128;
constexpr int WBYTES = 128 * 128;
constexpr int MT = 4;
constexpr int SLOT_BYTES = WBYTES + MT * 2048;
constexpr int PF = SLOTS - 1;
constexpr int CNT = 4 + MT / 2;
constexpr int LDS_BYTES = SLOTS * SLOT_BYTES + 64;
constexpr int NT_REM = (ML - XR) / 16;
constexpr int NPG = 8 - NRG;
constexpr int TPS = XR / 16;
constexpr int NPIECE = XR * 4 / (NPG * 32);
static_assert(XR % 16 == 0 && (XR * 4) % (NPG * 32) == 0, "XR");
static_assert(NT_REM <= NRG * MT && TPS <= (NPG + 1) * MT, "tiles");
static_assert(LDS_BYTES <= 160 * 1024 && LDS_BYTES > 80 * 1024, "one workgroup per CU");
static_assert((PF - 1) * CNT <= 63 && PF >= 2 && PF <= 5, "vmcnt");

constexpr size_t WS_CALL = 0;
constexpr size_t WS_XCNT = 2048 * 128;
constexpr size_t WS_OCNT = WS_XCNT + 8 * 128;
constexpr size_t WS_TMO = WS_OCNT + 8 * 128;
constexpr size_t WS_DBG = 512 * 1024;
constexpr size_t WS_XS = 1u << 20;
constexpr size_t XS_BYTES = (size_t)7 * XR * KD;
constexpr size_t WS_NEED = WS_XS + 2 * XS_BYTES;
constexpr unsigned long long DEADLINE = 100000000ull;

struct Params {
  const unsigned char* x;
  const unsigned char* w;
  const float* sx;
  const float* sw;
  float* out[NR];
  unsigned char* ws[NR];
  int rank;
};

#define RSRC(p, n) __builtin_amdgcn_make_buffer_rsrc((void*)(p), (short)0, (int)(n), 0x00020000)
typedef __amdgpu_buffer_rsrc_t rsrc_t;

__device__ __forceinline__ unsigned long long now() { return __builtin_amdgcn_s_memrealtime(); }

template <int N> __device__ __forceinline__ void wait_vm_barrier() {
  asm volatile("s_waitcnt vmcnt(%0)\n\ts_barrier" ::"n"(N) : "memory");
}
__device__ __forceinline__ void wait_lgkm_barrier() { asm volatile("s_waitcnt lgkmcnt(0)\n\ts_barrier" ::: "memory"); }
template <int N> __device__ __forceinline__ void wait_vm() { asm volatile("s_waitcnt vmcnt(%0)" ::"n"(N) : "memory"); }

__device__ __forceinline__ v4u srd(const void* p, unsigned bytes) {
  const unsigned long long a = (unsigned long long)p;
  return v4u{(unsigned)a, (unsigned)(a >> 32) & 0xffffu, bytes, 0x00020000u};
}
__device__ __forceinline__ void dma16(v4u r, unsigned dst, unsigned voff, unsigned soff) {
  asm volatile("s_mov_b32 m0, %0\n\ts_nop 0\n\tbuffer_load_dwordx4 %1, %2, %3 offen lds"
               ::"s"(dst), "v"(voff), "s"(r), "s"(soff) : "memory");
}

#define STAMP(idx) do {} while (0)

struct Loader {
  v4u wr, xr;
  unsigned v0, v1, kw0, kx0, dst0;
  int krot;
  __device__ __forceinline__ void init(lds_char* lds, const unsigned char* xbase, unsigned xbytes, unsigned xrow0, int nt,
                                       const unsigned char* wbase, int krot_, int wave, int lane) {
    const unsigned rowoff = (unsigned)(lane >> 3) * 4096u;
    const unsigned c0 = (unsigned)((lane & 7) ^ ((lane >> 4) & 3)) * 16u;
    v0 = rowoff + c0;
    v1 = rowoff + (c0 ^ 64u);
    wr = srd(wbase, (unsigned)(KD * ND - 3968));
    xr = srd(xbase, wave < nt ? xbytes : 0u);
    kw0 = (unsigned)wave * 32u * 4096u;
    kx0 = (xrow0 + (unsigned)wave * 16u) * 4096u;
    dst0 = (unsigned)(unsigned long long)lds;
    krot = krot_;
    dst0 += (unsigned)wave * 4096u;
    wv = wave;
  }
  int wv;
  __device__ __forceinline__ void issue(int k, unsigned so) {
    const unsigned kk = (unsigned)(k + krot) & (NSTEP - 1);
    const unsigned d = dst0 + so, kw = kw0 + kk * (128u * 4096u), kx = kx0 + kk * 128u;
    dma16(wr, d, v0, kw);
    dma16(wr, d + 1024, v0, kw + 8 * 4096);
    dma16(wr, d + 2048, v1, kw + 16 * 4096);
    dma16(wr, d + 3072, v1, kw + 24 * 4096);
    const unsigned dx = d + WBYTES - (unsigned)wv * 2048u;
    dma16(xr, dx, v0, kx);
    dma16(xr, dx + 1024, v1, kx + 8 * 4096);
  }
  __device__ __forceinline__ void prologue() {
#pragma unroll
    for (int s = 0; s < PF; ++s) issue(s, s * SLOT_BYTES);
  }
  __device__ __forceinline__ void run() {
    unsigned prev = PF * SLOT_BYTES;
#pragma unroll 1
    for (int k = 0; k < NSTEP - PF; ++k) {
      wait_vm_barrier<(PF - 1) * CNT>();
      issue(k + PF, prev);
      prev = (prev + SLOT_BYTES == SLOTS * SLOT_BYTES) ? 0u : prev + SLOT_BYTES;
    }
    wait_vm_barrier<(PF - 1) * CNT>();
    if (PF > 2) wait_vm_barrier<(PF > 2 ? PF - 2 : 0) * CNT>();
    if (PF > 3) wait_vm_barrier<(PF > 3 ? PF - 3 : 0) * CNT>();
    if (PF > 4) wait_vm_barrier<(PF > 4 ? PF - 4 : 0) * CNT>();
    wait_vm_barrier<0>();
  }
};

__device__ __forceinline__ void mma_tile(lds_char* lds, int wave, int lane, v4f (&acc)[MT][2], unsigned long long* ts) {
  const int g = lane >> 4;
  const unsigned ax0 = (unsigned)((lane & 15) * 128 + ((g ^ ((lane >> 1) & 7)) * 16)) + WBYTES;
  const unsigned ax1 = ax0 ^ 64u;
  const int q = (lane & 15) >> 1;
  const unsigned aw0 = (unsigned)((16 * g + q) * 128 + (((2 * wave) ^ (((q >> 1) & 3) | ((g & 1) << 2))) * 16) + 8 * (lane & 1));
  const unsigned aw1 = aw0 ^ 16u;
  auto rd = [&](unsigned so, v8i (&xf)[MT], v8i (&wf)[2]) {
#pragma unroll
    for (int u = 0; u < 2; ++u) {
      lds_char* a = lds + ((u ? aw1 : aw0) + so);
      v2i r0 = __builtin_amdgcn_ds_read_tr8_b64_v2i32((__attribute__((address_space(3))) v2i*)(a));
      v2i r1 = __builtin_amdgcn_ds_read_tr8_b64_v2i32((__attribute__((address_space(3))) v2i*)(a + 1024));
      v2i r2 = __builtin_amdgcn_ds_read_tr8_b64_v2i32((__attribute__((address_space(3))) v2i*)(a + 8192));
      v2i r3 = __builtin_amdgcn_ds_read_tr8_b64_v2i32((__attribute__((address_space(3))) v2i*)(a + 9216));
      wf[u] = v8i{r0[0], r0[1], r1[0], r1[1], r2[0], r2[1], r3[0], r3[1]};
    }
    lds_char* a0 = lds + (ax0 + so);
    lds_char* a1 = lds + (ax1 + so);
#pragma unroll
    for (int t = 0; t < MT; ++t) {
      v4i lo = *(__attribute__((address_space(3))) v4i*)(a0 + t * 2048);
      v4i hi = *(__attribute__((address_space(3))) v4i*)(a1 + t * 2048);
      xf[t] = v8i{lo[0], lo[1], lo[2], lo[3], hi[0], hi[1], hi[2], hi[3]};
    }
  };
  auto mma = [&](v8i (&xf)[MT], v8i (&wf)[2]) {
#pragma unroll
    for (int t = 0; t < MT; ++t)
#pragma unroll
      for (int u = 0; u < 2; ++u)
        acc[t][u] = __builtin_amdgcn_mfma_scale_f32_16x16x128_f8f6f4(wf[u], xf[t], acc[t][u], 0, 0, 0, 0, 0, 0);
  };
#pragma unroll
  for (int t = 0; t < MT; ++t) acc[t][0] = acc[t][1] = v4f{0.f, 0.f, 0.f, 0.f};
  v8i xa[MT], wa[2], xb[MT], wb[2];
#pragma unroll
  for (int t = 0; t < MT; ++t) xb[t] = v8i{0, 0, 0, 0, 0, 0, 0, 0};
  wb[0] = wb[1] = v8i{0, 0, 0, 0, 0, 0, 0, 0};
  unsigned cur = 0;
#define SB __builtin_amdgcn_sched_barrier(0)
#define NEXT cur = (cur + SLOT_BYTES == SLOTS * SLOT_BYTES) ? 0u : cur + SLOT_BYTES
#define MIX do { __builtin_amdgcn_sched_group_barrier(0x002, 4, 0); \
    _Pragma("unroll") for (int i = 0; i < 2 * MT; ++i) { \
      __builtin_amdgcn_sched_group_barrier(0x008, 1, 0); __builtin_amdgcn_sched_group_barrier(0x100, 2, 0); } SB; } while (0)
#define STEP(XN, WN, XO, WO) do { rd(cur, XN, WN); mma(XO, WO); MIX; NEXT; } while (0)
#pragma unroll 1
  for (int k = 0; k < NSTEP; k += 2) {
    wait_lgkm_barrier();
    STEP(xa, wa, xb, wb);
    wait_lgkm_barrier();
    STEP(xb, wb, xa, wa);
  }
  mma(xb, wb);
}

__device__ __forceinline__ void store_tile(rsrc_t dst, int nt, unsigned so, bool remote, float s, int lane, v4f (&acc)[MT][2]) {
  const unsigned voff = (unsigned)(lane & 7) * 2048u + (unsigned)(lane & 8) * 8u + (unsigned)(lane >> 4) * 16u;
#pragma unroll
  for (int t = 0; t < MT; ++t) {
    if (t < nt) {
      v4u o1, o2;
#pragma unroll
      for (int r = 0; r < 4; ++r) {
        const int a = __builtin_bit_cast(int, acc[t][0][r] * s), b = __builtin_bit_cast(int, acc[t][1][r] * s);
        o1[r] = (unsigned)__builtin_amdgcn_update_dpp(a, b, 0x118, 0xf, 0xc, false);
        o2[r] = (unsigned)__builtin_amdgcn_update_dpp(b, a, 0x108, 0xf, 0x3, false);
      }
      if (remote) { if (2 * (nt - t) <= TAILN) wait_vm<TAILW - 1>(); else wait_vm<OWIN - 1>(); }
      __builtin_amdgcn_raw_buffer_store_b128(o1, dst, voff, so + (unsigned)t * 16u * 2048u, 17);
      if (remote) { if (2 * (nt - t) - 1 <= TAILN) wait_vm<TAILW - 1>(); else wait_vm<OWIN - 1>(); }
      __builtin_amdgcn_raw_buffer_store_b128(o2, dst, voff, so + (unsigned)t * 16u * 2048u + 8u * 2048u, 17);
    }
  }
}

__device__ __forceinline__ bool wait_counters(const unsigned char* myws, size_t base, bool need, unsigned target,
                                              unsigned call, int lane) {
  unsigned* tmo = (unsigned*)(myws + WS_TMO);
  const unsigned* c = (const unsigned*)(myws + base + (size_t)(lane & 7) * 128);
  const unsigned long long t0 = now();
  for (;;) {
    const unsigned v = __hip_atomic_load(c, __ATOMIC_RELAXED, __HIP_MEMORY_SCOPE_SYSTEM);
    if (__all(!need || (int)(v - target) >= 0)) return true;
    if (__hip_atomic_load(tmo, __ATOMIC_RELAXED, __HIP_MEMORY_SCOPE_AGENT) == call || now() - t0 > DEADLINE) {
      if (lane == 0) __hip_atomic_store(tmo, call, __ATOMIC_RELAXED, __HIP_MEMORY_SCOPE_AGENT);
      return false;
    }
    __builtin_amdgcn_s_sleep(1);
  }
}

}

__global__ void __launch_bounds__(512, 2) gemm_a2a_kernel(Params P) {
  extern __shared__ __attribute__((aligned(16))) char smem[];
  lds_char* lds = (lds_char*)smem;
  const int lane = threadIdx.x & 63;
  const int wave8 = __builtin_amdgcn_readfirstlane(threadIdx.x >> 6);
  const int wave = wave8 & 3;
  const int b = blockIdx.x, lab = b & 7, idx = b >> 3, ct = idx & 3, rg = idx >> 2;
  const int rank = P.rank & 7;
  unsigned char* myws = P.ws[0];
  unsigned char* dws = P.ws[lab];
  float* myout = P.out[0];
  float* dout = P.out[lab];
  const unsigned char* px = P.x;
  const unsigned char* pw = P.w;
  const float* psx = P.sx;
  const float* psw = P.sw;
  asm volatile("" ::"s"(rank), "s"(myws), "s"(dws), "s"(myout), "s"(dout), "s"(px), "s"(pw), "s"(psx), "s"(psw));
  const bool pusher = lab != 0 && rg >= NRG;
  const int t1 = lab == 0 ? rg * 4 : (NT_REM * rg) / NRG;
  const int n1 = lab == 0 ? 4 : (NT_REM * (rg + 1)) / NRG - t1;
  const int krot = lab * ROT_L + ROT_RG * (lab ? (rg * 32 + NRG / 2) / NRG : rg * 4) + ct * ROT_CT;
  unsigned call = 0;
  const bool waiter = b == 28 * 8 && wave8 == 0;
  if (pusher || waiter || (lab == 0 && wave8 == 4))
    call = __hip_atomic_fetch_add((unsigned*)(myws + WS_CALL + (size_t)(b * 8 + wave8) * 128) + (lane & 31), 1u, __ATOMIC_RELAXED, __HIP_MEMORY_SCOPE_AGENT);
  Loader ld;
  if (wave8 >= 4 && !pusher) {
    ld.init(lds, px, ML * KD, (unsigned)t1 * 16u, n1, pw + ((rank + lab) & 7) * ML + ct * 128, krot, wave, lane);
    ld.prologue();
    ld.run();
    if (lab != 0 || rg == 7) return;
  }
  const int part2 = pusher ? rg - NRG : NPG;
  const int slot2 = pusher ? lab - 1 : rg;
  const int t2 = (TPS * part2) / (NPG + 1);
  const int n2 = (TPS * (part2 + 1)) / (NPG + 1) - t2;
  __attribute__((address_space(3))) int* flag = (__attribute__((address_space(3))) int*)(lds + LDS_BYTES - 64);
  float sxv = 0.f, swv = 0.f;
  if (wave8 < 4) { sxv = psx[0]; swv = psw[0]; }

  if (pusher) {
    const rsrc_t xs = RSRC(px + (size_t)(ML - XR) * KD, XR * KD);
    const unsigned off = (unsigned)((idx - NRG * 4) * 8 + wave8) * 1024u + (unsigned)lane * 16u;
    v4u d[NPIECE];
#pragma unroll
    for (int i = 0; i < NPIECE; ++i) d[i] = __builtin_amdgcn_raw_buffer_load_b128(xs, off, i * (NPG * 32768), 0);
    call = (__builtin_amdgcn_readfirstlane(call) >> 1) + 1u;
    const rsrc_t xd = RSRC(dws + WS_XS + (size_t)(call & 1u) * XS_BYTES + (size_t)(7 - lab) * XR * KD, XR * KD);
#pragma unroll
    for (int i = 0; i < NPIECE; ++i) {
      if (i >= PWIN) wait_vm<PWIN - 1>();
      __builtin_amdgcn_raw_buffer_store_b128(d[i], xd, off, i * (NPG * 32768), 17);
    }
    wait_vm<0>();
    if (wave8 == 0) STAMP(4);
    if (lane == 0) __hip_atomic_fetch_add((unsigned*)(dws + WS_XCNT + (size_t)rank * 128), 1u, __ATOMIC_RELAXED, __HIP_MEMORY_SCOPE_SYSTEM);
  } else {
    call = (__builtin_amdgcn_readfirstlane(call) >> 1) + 1u;
  }
  asm volatile("" : "+v"(sxv), "+v"(swv));
  float scale = sxv * swv;
  const unsigned colbyte = (unsigned)(ct * 128 + wave * 32) * 4u;

#pragma unroll 1
  for (int job = (pusher || wave8 >= 4) ? 1 : 0; job < ((lab == 0 ? rg < 7 : pusher) ? 2 : 1); ++job) {
    unsigned orow = (unsigned)(rank * ML + t1 * 16);
    int nt = n1;
    float* obase = dout;
    if (job == 1) {
      const int src = (rank + 1 + slot2) & 7;
      if (wave8 == 4) {
        const bool ok = wait_counters(myws, WS_XCNT, lane == src, call * (NPG * 32u), call, lane);
        __builtin_amdgcn_fence(__ATOMIC_ACQUIRE, "");
        wait_vm<0>();
        if (lane == 0) { flag[0] = ok ? 1 : 0; flag[1] = (int)call; }
      }
      wait_lgkm_barrier();
      const bool ok = __builtin_amdgcn_readfirstlane(flag[0]) != 0;
      call = (unsigned)__builtin_amdgcn_readfirstlane(flag[1]);
      if (wave8 >= 4) {
        ld.init(lds, myws + WS_XS + (size_t)(call & 1u) * XS_BYTES, ok ? (unsigned)XS_BYTES : 0u, (unsigned)(slot2 * XR + t2 * 16), n2,
                pw + rank * ML + ct * 128, pusher ? lab * ROT2_L + part2 * (32 / NPG) + ct * ROT2_CT : krot, wave, lane);
        ld.prologue();
        ld.run();
        return;
      }
      if (wave8 == 0) STAMP(3);
      nt = n2;
      obase = myout;
      orow = (unsigned)(src * ML + (ML - XR) + t2 * 16);
      if (!ok) scale = __builtin_nanf("");
    }
    v4f acc[MT][2];
    mma_tile(lds, wave, lane, acc, nullptr);
    store_tile(RSRC(obase, NR * ML * ML * 4), nt, orow * 2048u + colbyte, job == 0 && lab != 0, scale, lane, acc);
  }
  if (lab != 0 && !pusher) {
    wait_vm<0>();
    if (wave == 0) STAMP(2);
    if (lane == 0) __hip_atomic_fetch_add((unsigned*)(dws + WS_OCNT + (size_t)rank * 128), 1u, __ATOMIC_RELAXED, __HIP_MEMORY_SCOPE_SYSTEM);
    return;
  }
  if (wave == 0) STAMP(2);
  if (waiter) {
    wait_vm<0>();
    const bool ok = wait_counters(myws, WS_OCNT, lane < 8 && lane != rank, call * (NRG * 16u), call, lane);
    if (!ok && lane == 0) myout[(size_t)(rank * ML + 448) * ML] = __builtin_nanf("");
    STAMP(5);
  }
}

extern "C" void kernel_launch(int rank, int nranks, ncclComm_t comm, hipStream_t stream,
                              void* const* d_in, const int* in_sizes, int n_in,
                              void* d_out, int out_size,
                              const int* in_dtypes, int out_dtype,
                              void* const* d_out_all, void* const* d_ws_all, size_t ws_bytes) {
  (void)comm; (void)in_dtypes; (void)out_dtype; (void)d_out;
  if (nranks != NR || n_in != 4 || in_sizes[0] != ML * KD || in_sizes[1] != KD * ND || out_size != NR * ML * ML ||
      ws_bytes < WS_NEED || (unsigned)rank >= (unsigned)NR) {
    fprintf(stderr, "kernel_launch: unexpected problem shape\n");
    return;
  }
  static bool init = false;
  if (!init) {
    HIP_CHECK(hipFuncSetAttribute((const void*)gemm_a2a_kernel, hipFuncAttributeMaxDynamicSharedMemorySize, LDS_BYTES));
    init = true;
  }
  Params P;
  P.x = (const unsigned char*)d_in[0];
  P.w = (const unsigned char*)d_in[1];
  P.sx = (const float*)d_in[2];
  P.sw = (const float*)d_in[3];
  for (int p = 0; p < NR; ++p) {
    P.out[p] = (float*)d_out_all[(rank + p) % NR];
    P.ws[p] = (unsigned char*)d_ws_all[(rank + p) % NR];
  }
  P.rank = rank;
  hipLaunchKernelGGL(gemm_a2a_kernel, dim3(256), dim3(512), LDS_BYTES, stream, P);
  HIP_CHECK(hipGetLastError());
}
```

```cpp
#include <hip/hip_runtime.h>
#include <rccl/rccl.h>
#include <cstdio>

#define HIP_CHECK(expr) do { hipError_t _e = (expr); if (_e != hipSuccess) { \
    fprintf(stderr, "HIP error %s at %s:%d\n", hipGetErrorString(_e), __FILE__, __LINE__); \
    return; } } while (0)

#define XR 128
#define NRG 6
#define OWIN 1
#define PWIN 1
#define TAILW 2
#define TAILN 2
#define ROT_L 4
#define ROT_RG 1
#define ROT_CT 0
#define ROT2_L 2
#define ROT2_CT 0
#define PFIRST 1
#define L0DELAY 300
#define CTDELAY 0
#define CTSH 1
#define PJIT 2
#define ROT_G 0
#define LPRIO 0
#define EARLY_ACQ 1
#define POLLN 1
#define SLOTS 4
#define ILV 1

namespace {

typedef int v2i __attribute__((ext_vector_type(2)));
typedef int v4i __attribute__((ext_vector_type(4)));
typedef int v8i __attribute__((ext_vector_type(8)));
typedef unsigned v4u __attribute__((ext_vector_type(4)));
typedef float v4f __attribute__((ext_vector_type(4)));
typedef __attribute__((address_space(3))) char lds_char;

constexpr int NR = 8;
constexpr int KD = 4096;
constexpr int ND = 4096;
constexpr int ML = 512;
constexpr int NSTEP = KD / 128;
constexpr int WBYTES = 128 * 128;
constexpr int MT = 4;
constexpr int SLOT_BYTES = WBYTES + MT * 2048;
constexpr int PF = SLOTS - 1;
constexpr int CNT = 4 + MT / 2;
constexpr int LDS_BYTES = SLOTS * SLOT_BYTES + 64;
constexpr int NT_REM = (ML - XR) / 16;
constexpr int NPG = 8 - NRG;
constexpr int TPS = XR / 16;
constexpr int NPIECE = XR * 4 / (NPG * 32);
static_assert(XR % 16 == 0 && (XR * 4) % (NPG * 32) == 0, "XR");
static_assert(NPG * 32 <= 64 && NRG * 16 <= 128, "flags");
static_assert(NT_REM <= NRG * MT && TPS <= (NPG + 1) * MT, "tiles");
static_assert(LDS_BYTES <= 160 * 1024 && LDS_BYTES > 80 * 1024, "one workgroup per CU");
static_assert((PF - 1) * CNT <= 63 && PF >= 2 && PF <= 5, "vmcnt");

constexpr size_t WS_CALL = 0;
constexpr size_t WS_XF = 2048 * 128;
constexpr size_t WS_OF = WS_XF + 8 * 256;
constexpr size_t WS_TMO = WS_OF + 8 * 512;
constexpr size_t WS_DBG = 512 * 1024;
constexpr size_t WS_XS = 1u << 20;
constexpr size_t XS_BYTES = (size_t)7 * XR * KD;
constexpr size_t WS_NEED = WS_XS + 2 * XS_BYTES;
constexpr unsigned long long DEADLINE = 100000000ull;

struct Params {
  const unsigned char* x;
  const unsigned char* w;
  const float* sx;
  const float* sw;
  float* out[NR];
  unsigned char* ws[NR];
  int rank;
};

#define RSRC(p, n) __builtin_amdgcn_make_buffer_rsrc((void*)(p), (short)0, (int)(n), 0x00020000)
typedef __amdgpu_buffer_rsrc_t rsrc_t;

__device__ __forceinline__ unsigned long long now() { return __builtin_amdgcn_s_memrealtime(); }

template <int N> __device__ __forceinline__ void wait_vm_barrier() {
  asm volatile("s_waitcnt vmcnt(%0)\n\ts_barrier" ::"n"(N) : "memory");
}
__device__ __forceinline__ void wait_lgkm_barrier() { asm volatile("s_waitcnt lgkmcnt(0)\n\ts_barrier" ::: "memory"); }
template <int N> __device__ __forceinline__ void wait_vm() { asm volatile("s_waitcnt vmcnt(%0)" ::"n"(N) : "memory"); }

__device__ __forceinline__ v4u srd(const void* p, unsigned bytes) {
  const unsigned long long a = (unsigned long long)p;
  return v4u{(unsigned)a, (unsigned)(a >> 32) & 0xffffu, bytes, 0x00020000u};
}
__device__ __forceinline__ void dma16(v4u r, unsigned dst, unsigned voff, unsigned soff) {
  asm volatile("s_mov_b32 m0, %0\n\ts_nop 0\n\tbuffer_load_dwordx4 %1, %2, %3 offen lds"
               ::"s"(dst), "v"(voff), "s"(r), "s"(soff) : "memory");
}

__device__ __forceinline__ void store16(v4u v, rsrc_t r, unsigned voff) { __builtin_amdgcn_raw_buffer_store_b128(v, r, voff, 0, 17); }

#define STAMP(idx) do {} while (0)

struct Loader {
  v4u wr, xr;
  unsigned v0, v1, kw0, kx0, dst0, dstx;
  int krot;
  __device__ __forceinline__ void init_w(lds_char* lds, const unsigned char* wbase, int krot_, int wave, int lane) {
    const unsigned rowoff = (unsigned)(lane >> 3) * 4096u;
    const unsigned c0 = (unsigned)((lane & 7) ^ ((lane >> 4) & 3)) * 16u;
    v0 = rowoff + c0;
    v1 = rowoff + (c0 ^ 64u);
    wr = srd(wbase, (unsigned)(KD * ND - 3968));
    kw0 = (unsigned)wave * 32u * 4096u;
    dst0 = (unsigned)(unsigned long long)lds + (unsigned)wave * 4096u;
    dstx = (unsigned)(unsigned long long)lds + WBYTES + (unsigned)wave * 2048u;
    krot = krot_;
    asm volatile("s_nop 4" ::"s"(wr), "s"(kw0), "s"(dst0), "s"(dstx), "s"(krot));
  }
  __device__ __forceinline__ void init_x(const unsigned char* xbase, unsigned xbytes, unsigned xrow0, int nt, int wave) {
    xr = srd(xbase, wave < nt ? xbytes : 0u);
    kx0 = (xrow0 + (unsigned)wave * 16u) * 4096u;
    asm volatile("s_nop 4" ::"s"(xr), "s"(kx0));
  }
  __device__ __forceinline__ void issue_w(int k, unsigned so) {
    const unsigned kw = kw0 + ((unsigned)(k + krot) & (NSTEP - 1)) * (128u * 4096u), d = dst0 + so;
    dma16(wr, d, v0, kw);
    dma16(wr, d + 1024, v0, kw + 8 * 4096);
    dma16(wr, d + 2048, v1, kw + 16 * 4096);
    dma16(wr, d + 3072, v1, kw + 24 * 4096);
  }
  __device__ __forceinline__ void issue_x(int k, unsigned so) {
    const unsigned kx = kx0 + ((unsigned)(k + krot) & (NSTEP - 1)) * 128u, d = dstx + so;
    dma16(xr, d, v0, kx);
    dma16(xr, d + 1024, v1, kx + 8 * 4096);
  }
  template <bool SPLIT> __device__ __forceinline__ void run() {
    unsigned prev = PF * SLOT_BYTES;
    if (SPLIT) {
#pragma unroll
      for (int k = 0; k < PF - 1; ++k) {
        if (k == 0) wait_vm_barrier<2 * (PF - 1)>();
        if (k == 1) wait_vm_barrier<2 * (PF - 2) + CNT>();
        if (k == 2) wait_vm_barrier<(PF > 3 ? 2 * (PF - 3) : 0) + 2 * CNT>();
        if (k == 3) wait_vm_barrier<(PF > 4 ? 2 * (PF - 4) : 0) + 3 * CNT>();
        issue_w(k + PF, prev);
        issue_x(k + PF, prev);
        prev = (prev + SLOT_BYTES == SLOTS * SLOT_BYTES) ? 0u : prev + SLOT_BYTES;
      }
    }
#pragma unroll 1
    for (int k = SPLIT ? PF - 1 : 0; k < NSTEP - PF; ++k) {
      wait_vm_barrier<(PF - 1) * CNT>();
      issue_w(k + PF, prev);
      issue_x(k + PF, prev);
      prev = (prev + SLOT_BYTES == SLOTS * SLOT_BYTES) ? 0u : prev + SLOT_BYTES;
    }
    wait_vm_barrier<(PF - 1) * CNT>();
    if (PF > 2) wait_vm_barrier<(PF > 2 ? PF - 2 : 0) * CNT>();
    if (PF > 3) wait_vm_barrier<(PF > 3 ? PF - 3 : 0) * CNT>();
    if (PF > 4) wait_vm_barrier<(PF > 4 ? PF - 4 : 0) * CNT>();
    wait_vm_barrier<0>();
  }
};

__device__ __forceinline__ void mma_tile(lds_char* lds, int wave, int lane, v4f (&acc)[MT][2], unsigned long long* ts) {
  const int g = lane >> 4;
  const unsigned ax0 = (unsigned)((lane & 15) * 128 + ((g ^ ((lane >> 1) & 7)) * 16)) + WBYTES;
  const unsigned ax1 = ax0 ^ 64u;
  const int q = (lane & 15) >> 1;
  const unsigned aw0 = (unsigned)((16 * g + q) * 128 + (((2 * wave) ^ (((q >> 1) & 3) | ((g & 1) << 2))) * 16) + 8 * (lane & 1));
  const unsigned aw1 = aw0 ^ 16u;
  auto rd = [&](unsigned so, v8i (&xf)[MT], v8i (&wf)[2]) {
#pragma unroll
    for (int u = 0; u < 2; ++u) {
      lds_char* a = lds + ((u ? aw1 : aw0) + so);
      v2i r0 = __builtin_amdgcn_ds_read_tr8_b64_v2i32((__attribute__((address_space(3))) v2i*)(a));
      v2i r1 = __builtin_amdgcn_ds_read_tr8_b64_v2i32((__attribute__((address_space(3))) v2i*)(a + 1024));
      v2i r2 = __builtin_amdgcn_ds_read_tr8_b64_v2i32((__attribute__((address_space(3))) v2i*)(a + 8192));
      v2i r3 = __builtin_amdgcn_ds_read_tr8_b64_v2i32((__attribute__((address_space(3))) v2i*)(a + 9216));
      wf[u] = v8i{r0[0], r0[1], r1[0], r1[1], r2[0], r2[1], r3[0], r3[1]};
    }
    lds_char* a0 = lds + (ax0 + so);
    lds_char* a1 = lds + (ax1 + so);
#pragma unroll
    for (int t = 0; t < MT; ++t) {
      v4i lo = *(__attribute__((address_space(3))) v4i*)(a0 + t * 2048);
      v4i hi = *(__attribute__((address_space(3))) v4i*)(a1 + t * 2048);
      xf[t] = v8i{lo[0], lo[1], lo[2], lo[3], hi[0], hi[1], hi[2], hi[3]};
    }
  };
  auto mma = [&](v8i (&xf)[MT], v8i (&wf)[2]) {
#pragma unroll
    for (int t = 0; t < MT; ++t)
#pragma unroll
      for (int u = 0; u < 2; ++u)
        acc[t][u] = __builtin_amdgcn_mfma_scale_f32_16x16x128_f8f6f4(wf[u], xf[t], acc[t][u], 0, 0, 0, 0, 0, 0);
  };
#pragma unroll
  for (int t = 0; t < MT; ++t) acc[t][0] = acc[t][1] = v4f{0.f, 0.f, 0.f, 0.f};
  v8i xa[MT], wa[2], xb[MT], wb[2];
#pragma unroll
  for (int t = 0; t < MT; ++t) xb[t] = v8i{0, 0, 0, 0, 0, 0, 0, 0};
  wb[0] = wb[1] = v8i{0, 0, 0, 0, 0, 0, 0, 0};
  unsigned cur = 0;
#define SB __builtin_amdgcn_sched_barrier(0)
#define NEXT cur = (cur + SLOT_BYTES == SLOTS * SLOT_BYTES) ? 0u : cur + SLOT_BYTES
#define MIX do { __builtin_amdgcn_sched_group_barrier(0x002, 4, 0); \
    _Pragma("unroll") for (int i = 0; i < 2 * MT; ++i) { \
      __builtin_amdgcn_sched_group_barrier(0x008, 1, 0); __builtin_amdgcn_sched_group_barrier(0x100, 2, 0); } SB; } while (0)
#define STEP(XN, WN, XO, WO) do { rd(cur, XN, WN); mma(XO, WO); MIX; NEXT; } while (0)
#pragma unroll 1
  for (int k = 0; k < NSTEP; k += 2) {
    wait_lgkm_barrier();
    STEP(xa, wa, xb, wb);
    wait_lgkm_barrier();
    STEP(xb, wb, xa, wa);
  }
  mma(xb, wb);
}

__device__ __forceinline__ void store_tile(rsrc_t dst, int nt, unsigned so, bool remote, float s, int lane, v4f (&acc)[MT][2]) {
  const unsigned voff = so + (unsigned)(lane & 7) * 2048u + (unsigned)(lane & 8) * 8u + (unsigned)(lane >> 4) * 16u;
#pragma unroll
  for (int t = 0; t < MT; ++t) {
    if (t < nt) {
      v4u o1, o2;
#pragma unroll
      for (int r = 0; r < 4; ++r) {
        const int a = __builtin_bit_cast(int, acc[t][0][r] * s), b = __builtin_bit_cast(int, acc[t][1][r] * s);
        o1[r] = (unsigned)__builtin_amdgcn_update_dpp(a, b, 0x118, 0xf, 0xc, false);
        o2[r] = (unsigned)__builtin_amdgcn_update_dpp(b, a, 0x108, 0xf, 0x3, false);
      }
      if (remote) { if (2 * (nt - t) <= TAILN) wait_vm<TAILW - 1>(); else wait_vm<OWIN - 1>(); }
      store16(o1, dst, voff + (unsigned)t * 16u * 2048u);
      if (remote) { if (2 * (nt - t) - 1 <= TAILN) wait_vm<TAILW - 1>(); else wait_vm<OWIN - 1>(); }
      store16(o2, dst, voff + (unsigned)t * 16u * 2048u + 8u * 2048u);
    }
  }
}

template <int W>
__device__ __forceinline__ bool wait_flags(const unsigned char* myws, unsigned off, bool need, unsigned call, int lane) {
  unsigned* tmo = (unsigned*)(myws + WS_TMO);
  const rsrc_t r = RSRC(myws, WS_TMO + 128);
  const unsigned long long t0 = now();
  for (;;) {
    bool ok;
    if (W == 1) {
      ok = (int)(__builtin_amdgcn_raw_buffer_load_b32(r, off, 0, 17) - call) >= 0;
    } else {
      const v4u v = __builtin_amdgcn_raw_buffer_load_b128(r, off, 0, 17);
      ok = (int)(v[0] - call) >= 0 && (int)(v[1] - call) >= 0 && (int)(v[2] - call) >= 0 && (int)(v[3] - call) >= 0;
    }
    const unsigned t = __hip_atomic_load(tmo, __ATOMIC_RELAXED, __HIP_MEMORY_SCOPE_AGENT);
    asm volatile("" ::: "memory");
    if (__all(!need || ok)) return true;
    if (t == call || now() - t0 > DEADLINE) {
      if (lane == 0) __hip_atomic_store(tmo, call, __ATOMIC_RELAXED, __HIP_MEMORY_SCOPE_AGENT);
      return false;
    }
    __builtin_amdgcn_s_sleep(1);
  }
}

}

__global__ void __launch_bounds__(512, 2) gemm_a2a_kernel(Params P) {
  extern __shared__ __attribute__((aligned(16))) char smem[];
  lds_char* lds = (lds_char*)smem;
  const int lane = threadIdx.x & 63;
  const int wave8 = __builtin_amdgcn_readfirstlane(threadIdx.x >> 6);
  const int wave = wave8 & 3;
  const int b = blockIdx.x, lab = b & 7, idx = b >> 3, ct = idx & 3;
  const int rg = ((idx >> 2) + PFIRST * NRG) & 7;
  const int rank = P.rank & 7;
  unsigned char* myws = P.ws[0];
  unsigned char* dws = P.ws[lab];
  float* myout = P.out[0];
  float* dout = P.out[lab];
  const unsigned char* px = P.x;
  const unsigned char* pw = P.w;
  const float* psx = P.sx;
  const float* psw = P.sw;
  asm volatile("" ::"s"(rank), "s"(myws), "s"(dws), "s"(myout), "s"(dout), "s"(px), "s"(pw), "s"(psx), "s"(psw));
  const bool pusher = lab != 0 && rg >= NRG;
  const int t1 = lab == 0 ? rg * 4 : (NT_REM * rg) / NRG;
  const int n1 = lab == 0 ? 4 : (NT_REM * (rg + 1)) / NRG - t1;
  const int krot = lab * ROT_L + ROT_RG * (lab ? (rg * 32 + NRG / 2) / NRG : rg * 4) + ct * ROT_CT;
  const bool waiter = lab == 0 && rg == 7 && ct == 0;
  const bool counts = pusher || wave8 < 4 || (lab == 0 && wave8 == 4);
  unsigned* cword = (unsigned*)(myws + WS_CALL + (size_t)(b * 8 + wave8) * 128);
  unsigned call = 0;
  float sxv = 0.f, swv = 0.f;
  Loader ld;
  if (pusher) {
    const rsrc_t xs = RSRC(px + (size_t)(ML - XR) * KD, XR * KD);
    const unsigned off = (unsigned)((rg - NRG) * 4 + ct) * 1024u + (unsigned)lane * 16u;
    __attribute__((address_space(3))) unsigned* next = (__attribute__((address_space(3))) unsigned*)(lds + LDS_BYTES - 32);
    unsigned piece = (unsigned)wave8;
    v4u d = __builtin_amdgcn_raw_buffer_load_b128(xs, off, piece * (NPG * 4096u), 0);
    call = __hip_atomic_load(cword, __ATOMIC_RELAXED, __HIP_MEMORY_SCOPE_SYSTEM);
    if (wave8 < 4) { sxv = psx[0]; swv = psw[0]; }
    if (threadIdx.x == 0) *next = 8u;
    wait_lgkm_barrier();
    call = __builtin_amdgcn_readfirstlane(call) + 1u;
    const rsrc_t xd = RSRC(dws + WS_XS + (size_t)(call & 1u) * XS_BYTES + (size_t)(7 - lab) * XR * KD, XR * KD);
    for (;;) {
      store16(d, xd, off + piece * (NPG * 4096u));
      unsigned nx = 0;
      if (lane == 0) nx = __hip_atomic_fetch_add(next, 1u, __ATOMIC_RELAXED, __HIP_MEMORY_SCOPE_WORKGROUP);
      piece = __builtin_amdgcn_readfirstlane(nx);
      if (piece >= 8u * NPIECE) break;
      d = __builtin_amdgcn_raw_buffer_load_b128(xs, off, piece * (NPG * 4096u), 0);
    }
    wait_vm<0>();
    if (lane == 0) __hip_atomic_store((unsigned*)(dws + WS_XF + (size_t)rank * 256) + (((rg - NRG) * 4 + ct) * 8 + wave8), call, __ATOMIC_RELAXED, __HIP_MEMORY_SCOPE_SYSTEM);
  } else {
    if (counts) call = __hip_atomic_load(cword, __ATOMIC_RELAXED, __HIP_MEMORY_SCOPE_SYSTEM);
    if (wave8 >= 4) {
      ld.init_w(lds, pw + ((rank + lab) & 7) * ML + ct * 128, krot, wave, lane);
      ld.init_x(px, ML * KD, (unsigned)t1 * 16u, n1, wave);
      if (L0DELAY > 0 || CTDELAY > 0) {
        const unsigned long long t = now();
        const unsigned dl = lab == 0 ? L0DELAY : (ct >> CTSH) * CTDELAY;
        while (now() - t < dl) __builtin_amdgcn_s_sleep(2);
      }
      if (LPRIO) __builtin_amdgcn_s_setprio(LPRIO);
#pragma unroll
      for (int s = 0; s < PF; ++s) { ld.issue_w(s, s * SLOT_BYTES); ld.issue_x(s, s * SLOT_BYTES); }
      ld.run<false>();
      if (lab != 0) return;
    } else {
      sxv = psx[0]; swv = psw[0];
    }
    call = __builtin_amdgcn_readfirstlane(call) + 1u;
  }
  if (counts && lane == 0) __hip_atomic_store(cword, call, __ATOMIC_RELAXED, __HIP_MEMORY_SCOPE_SYSTEM);
  if (wave8 >= 4 && lab == 0 && rg == 7) return;
  const int part2 = pusher ? rg - NRG : NPG;
  const int slot2 = pusher ? lab - 1 : rg;
  const int t2 = (TPS * part2) / (NPG + 1);
  const int n2 = (TPS * (part2 + 1)) / (NPG + 1) - t2;
  __attribute__((address_space(3))) int* flag = (__attribute__((address_space(3))) int*)(lds + LDS_BYTES - 64);
  asm volatile("" : "+v"(sxv), "+v"(swv));
  float scale = sxv * swv;
  const unsigned colbyte = (unsigned)(ct * 128 + wave * 32) * 4u;

#pragma unroll 1
  for (int job = (pusher || wave8 >= 4) ? 1 : 0; job < ((lab == 0 ? rg < 7 : pusher) ? 2 : 1); ++job) {
    unsigned orow = (unsigned)(rank * ML + t1 * 16);
    int nt = n1;
    float* obase = dout;
    if (job == 1) {
      const int src = (rank + 1 + slot2) & 7;
      if (wave8 >= 4) {
        ld.init_w(lds, pw + rank * ML + ct * 128, pusher ? lab * ROT2_L + part2 * (32 / NPG) + ct * ROT2_CT : krot, wave, lane);
#pragma unroll
        for (int s = 0; s < PF; ++s) ld.issue_w(s, s * SLOT_BYTES);
      }
      if (wave8 == 4) {
        __builtin_amdgcn_fence(__ATOMIC_ACQUIRE, "");
        const bool ok = wait_flags<1>(myws, (unsigned)WS_XF + (unsigned)src * 256u + (unsigned)lane * 4u, lane < NPG * 32, call, lane);
        wait_vm<0>();
        if (lane == 0) { flag[0] = ok ? 1 : 0; flag[1] = (int)call; }
      }
      wait_lgkm_barrier();
      const bool ok = __builtin_amdgcn_readfirstlane(flag[0]) != 0;
      call = (unsigned)__builtin_amdgcn_readfirstlane(flag[1]);
      if (wave8 >= 4) {
        ld.init_x(myws + WS_XS + (size_t)(call & 1u) * XS_BYTES, ok ? (unsigned)XS_BYTES : 0u, (unsigned)(slot2 * XR + t2 * 16), n2, wave);
#pragma unroll
        for (int s = 0; s < PF; ++s) ld.issue_x(s, s * SLOT_BYTES);
        ld.run<true>();
        return;
      }
      if (wave8 == 0) STAMP(3);
      nt = n2;
      obase = myout;
      orow = (unsigned)(src * ML + (ML - XR) + t2 * 16);
      if (!ok) scale = __builtin_nanf("");
    }
    v4f acc[MT][2];
    mma_tile(lds, wave, lane, acc, nullptr);
    store_tile(RSRC(obase, NR * ML * ML * 4), nt, orow * 2048u + colbyte, job == 0 && lab != 0, scale, lane, acc);
  }
  if (lab != 0 && !pusher) {
    wait_vm<0>();
    if (wave == 0) STAMP(2);
    if (lane == 0) __hip_atomic_store((unsigned*)(dws + WS_OF + (size_t)rank * 512) + ((rg * 4 + ct) * 4 + wave), call, __ATOMIC_RELAXED, __HIP_MEMORY_SCOPE_SYSTEM);
    return;
  }
  if (wave == 0) STAMP(2);
  if (waiter) {
    wait_vm<0>();
    const bool ok = wait_flags<4>(myws, (unsigned)WS_OF + (unsigned)wave * 1024u + (unsigned)lane * 16u,
                                  wave * 2 + (lane >> 5) != rank && (lane & 31) < NRG * 4, call, lane);
    if (!ok && lane == 0) myout[(size_t)(rank * ML + 448) * ML + wave * 32] = __builtin_nanf("");
  }
}

extern "C" void kernel_launch(int rank, int nranks, ncclComm_t comm, hipStream_t stream,
                              void* const* d_in, const int* in_sizes, int n_in,
                              void* d_out, int out_size,
                              const int* in_dtypes, int out_dtype,
                              void* const* d_out_all, void* const* d_ws_all, size_t ws_bytes) {
  (void)comm; (void)in_dtypes; (void)out_dtype; (void)d_out;
  if (nranks != NR || n_in != 4 || in_sizes[0] != ML * KD || in_sizes[1] != KD * ND || out_size != NR * ML * ML ||
      ws_bytes < WS_NEED || (unsigned)rank >= (unsigned)NR) {
    fprintf(stderr, "kernel_launch: unexpected problem shape\n");
    return;
  }
  static bool init = false;
  if (!init) {
    HIP_CHECK(hipFuncSetAttribute((const void*)gemm_a2a_kernel, hipFuncAttributeMaxDynamicSharedMemorySize, LDS_BYTES));
    init = true;
  }
  Params P;
  P.x = (const unsigned char*)d_in[0];
  P.w = (const unsigned char*)d_in[1];
  P.sx = (const float*)d_in[2];
  P.sw = (const float*)d_in[3];
  for (int p = 0; p < NR; ++p) {
    P.out[p] = (float*)d_out_all[(rank + p) % NR];
    P.ws[p] = (unsigned char*)d_ws_all[(rank + p) % NR];
  }
  P.rank = rank;
  hipLaunchKernelGGL(gemm_a2a_kernel, dim3(256), dim3(512), LDS_BYTES, stream, P);
  HIP_CHECK(hipGetLastError());
}
```

```cpp
#include <hip/hip_runtime.h>
#include <rccl/rccl.h>
#include <cstdio>

#define HIP_CHECK(expr) do { hipError_t _e = (expr); if (_e != hipSuccess) { \
    fprintf(stderr, "HIP error %s at %s:%d\n", hipGetErrorString(_e), __FILE__, __LINE__); \
    return; } } while (0)

#define XR 128
#define NRG 6
#define PWIN 2
#define TAILN 2
#define ROT_L 4
#define L0DELAY 300
#define PFIRST 1

namespace {

typedef int v2i __attribute__((ext_vector_type(2)));
typedef int v4i __attribute__((ext_vector_type(4)));
typedef int v8i __attribute__((ext_vector_type(8)));
typedef unsigned v4u __attribute__((ext_vector_type(4)));
typedef float v4f __attribute__((ext_vector_type(4)));
typedef __attribute__((address_space(3))) char lds_char;

constexpr int NR = 8;
constexpr int KD = 4096;
constexpr int ND = 4096;
constexpr int ML = 512;
constexpr int NSTEP = KD / 128;
constexpr int WBYTES = 128 * 128;
constexpr int MT = 4;
constexpr int SLOTS = 4;
constexpr int SLOT_BYTES = WBYTES + MT * 2048;
constexpr int PF = SLOTS - 1;
constexpr int CNT = 4 + MT / 2;
constexpr int LDS_BYTES = SLOTS * SLOT_BYTES + 64;
constexpr int NT_REM = (ML - XR) / 16;
constexpr int NPG = 8 - NRG;
constexpr int TPS = XR / 16;
constexpr int NPIECE = XR / NPG;
static_assert(XR % 16 == 0 && XR % NPG == 0 && NPIECE >= 8 * PWIN, "XR");
static_assert(NPG * 32 <= 64 && NRG * 16 <= 128, "flags");
static_assert(NT_REM <= NRG * MT && TPS <= (NPG + 1) * MT, "tiles");
static_assert(LDS_BYTES <= 160 * 1024 && LDS_BYTES > 80 * 1024, "one workgroup per CU");
static_assert(PF == 3, "counted waits");

constexpr size_t WS_CALL = 0;
constexpr size_t WS_XF = 2048 * 128;
constexpr size_t WS_OF = WS_XF + 8 * 256;
constexpr size_t WS_TMO = WS_OF + 8 * 512;
constexpr size_t WS_DBG = 512 * 1024;
constexpr size_t WS_XS = 1u << 20;
constexpr size_t XS_BYTES = (size_t)7 * XR * KD;
constexpr size_t WS_NEED = WS_XS + 2 * XS_BYTES;
constexpr unsigned long long DEADLINE = 100000000ull;

struct Params {
  const unsigned char* x;
  const unsigned char* w;
  const float* sx;
  const float* sw;
  float* out[NR];
  unsigned char* ws[NR];
  int rank;
};

#define RSRC(p, n) __builtin_amdgcn_make_buffer_rsrc((void*)(p), (short)0, (int)(n), 0x00020000)
typedef __amdgpu_buffer_rsrc_t rsrc_t;

__device__ __forceinline__ unsigned long long now() { return __builtin_amdgcn_s_memrealtime(); }

template <int N> __device__ __forceinline__ void wait_vm_barrier() {
  asm volatile("s_waitcnt vmcnt(%0)\n\ts_barrier" ::"n"(N) : "memory");
}
__device__ __forceinline__ void wait_lgkm_barrier() { asm volatile("s_waitcnt lgkmcnt(0)\n\ts_barrier" ::: "memory"); }
template <int N> __device__ __forceinline__ void wait_vm() { asm volatile("s_waitcnt vmcnt(%0)" ::"n"(N) : "memory"); }

__device__ __forceinline__ v4u srd(const void* p, unsigned bytes) {
  const unsigned long long a = (unsigned long long)p;
  return v4u{(unsigned)a, (unsigned)(a >> 32) & 0xffffu, bytes, 0x00020000u};
}
__device__ __forceinline__ void dma16(v4u r, unsigned dst, unsigned voff, unsigned soff) {
  asm volatile("s_mov_b32 m0, %0\n\ts_nop 0\n\tbuffer_load_dwordx4 %1, %2, %3 offen lds"
               ::"s"(dst), "v"(voff), "s"(r), "s"(soff) : "memory");
}
__device__ __forceinline__ void store16(v4u v, rsrc_t r, unsigned voff) { __builtin_amdgcn_raw_buffer_store_b128(v, r, voff, 0, 17); }
__device__ __forceinline__ void put_word(void* p, unsigned v) {
  __hip_atomic_store((unsigned*)p, v, __ATOMIC_RELAXED, __HIP_MEMORY_SCOPE_SYSTEM);
}

#define STAMP(idx) do {} while (0)
#define STAMPMAX(idx) do {} while (0)

struct Loader {
  v4u wr, xr;
  unsigned v0, v1, kw0, kx0, dst0, dstx;
  int krot;
  __device__ __forceinline__ void init_w(lds_char* lds, const unsigned char* wbase, int krot_, int wave, int lane) {
    const unsigned rowoff = (unsigned)(lane >> 3) * 4096u;
    const unsigned c0 = (unsigned)((lane & 7) ^ ((lane >> 4) & 3)) * 16u;
    v0 = rowoff + c0;
    v1 = rowoff + (c0 ^ 64u);
    wr = srd(wbase, (unsigned)(KD * ND - 3968));
    kw0 = (unsigned)wave * 32u * 4096u;
    dst0 = (unsigned)(unsigned long long)lds + (unsigned)wave * 4096u;
    dstx = (unsigned)(unsigned long long)lds + WBYTES + (unsigned)wave * 2048u;
    krot = krot_;
    asm volatile("s_nop 4" ::"s"(wr), "s"(kw0), "s"(dst0), "s"(dstx), "s"(krot));
  }
  __device__ __forceinline__ void init_x(const unsigned char* xbase, unsigned xbytes, unsigned xrow0, int nt, int wave) {
    xr = srd(xbase, wave < nt ? xbytes : 0u);
    kx0 = (xrow0 + (unsigned)wave * 16u) * 4096u;
    asm volatile("s_nop 4" ::"s"(xr), "s"(kx0));
  }
  __device__ __forceinline__ void issue_w(int k, unsigned so) {
    const unsigned kw = kw0 + ((unsigned)(k + krot) & (NSTEP - 1)) * (128u * 4096u), d = dst0 + so;
    dma16(wr, d, v0, kw);
    dma16(wr, d + 1024, v0, kw + 8 * 4096);
    dma16(wr, d + 2048, v1, kw + 16 * 4096);
    dma16(wr, d + 3072, v1, kw + 24 * 4096);
  }
  __device__ __forceinline__ void issue_x(int k, unsigned so) {
    const unsigned kx = kx0 + ((unsigned)(k + krot) & (NSTEP - 1)) * 128u, d = dstx + so;
    dma16(xr, d, v0, kx);
    dma16(xr, d + 1024, v1, kx + 8 * 4096);
  }
  template <bool SPLIT> __device__ __forceinline__ void run() {
    unsigned prev = PF * SLOT_BYTES;
    if (SPLIT) {
      wait_vm_barrier<4>();
      issue_w(3, 3 * SLOT_BYTES); issue_x(3, 3 * SLOT_BYTES);
      wait_vm_barrier<2 + CNT>();
      issue_w(4, 0); issue_x(4, 0);
      prev = SLOT_BYTES;
    }
#pragma unroll 1
    for (int k = SPLIT ? 2 : 0; k < NSTEP - PF; ++k) {
      wait_vm_barrier<2 * CNT>();
      issue_w(k + PF, prev);
      issue_x(k + PF, prev);
      prev = (prev + SLOT_BYTES == SLOTS * SLOT_BYTES) ? 0u : prev + SLOT_BYTES;
    }
    wait_vm_barrier<2 * CNT>();
    wait_vm_barrier<CNT>();
    wait_vm_barrier<0>();
  }
};

__device__ __forceinline__ void mma_tile(lds_char* lds, int wave, int lane, v4f (&acc)[MT][2]) {
  const int g = lane >> 4;
  const unsigned ax0 = (unsigned)((lane & 15) * 128 + ((g ^ ((lane >> 1) & 7)) * 16)) + WBYTES;
  const unsigned ax1 = ax0 ^ 64u;
  const int q = (lane & 15) >> 1;
  const unsigned aw0 = (unsigned)((16 * g + q) * 128 + (((2 * wave) ^ (((q >> 1) & 3) | ((g & 1) << 2))) * 16) + 8 * (lane & 1));
  const unsigned aw1 = aw0 ^ 16u;
  auto rd = [&](unsigned so, v8i (&xf)[MT], v8i (&wf)[2]) {
#pragma unroll
    for (int u = 0; u < 2; ++u) {
      lds_char* a = lds + ((u ? aw1 : aw0) + so);
      v2i r0 = __builtin_amdgcn_ds_read_tr8_b64_v2i32((__attribute__((address_space(3))) v2i*)(a));
      v2i r1 = __builtin_amdgcn_ds_read_tr8_b64_v2i32((__attribute__((address_space(3))) v2i*)(a + 1024));
      v2i r2 = __builtin_amdgcn_ds_read_tr8_b64_v2i32((__attribute__((address_space(3))) v2i*)(a + 8192));
      v2i r3 = __builtin_amdgcn_ds_read_tr8_b64_v2i32((__attribute__((address_space(3))) v2i*)(a + 9216));
      wf[u] = v8i{r0[0], r0[1], r1[0], r1[1], r2[0], r2[1], r3[0], r3[1]};
    }
    lds_char* a0 = lds + (ax0 + so);
    lds_char* a1 = lds + (ax1 + so);
#pragma unroll
    for (int t = 0; t < MT; ++t) {
      v4i lo = *(__attribute__((address_space(3))) v4i*)(a0 + t * 2048);
      v4i hi = *(__attribute__((address_space(3))) v4i*)(a1 + t * 2048);
      xf[t] = v8i{lo[0], lo[1], lo[2], lo[3], hi[0], hi[1], hi[2], hi[3]};
    }
  };
  auto mma = [&](v8i (&xf)[MT], v8i (&wf)[2]) {
#pragma unroll
    for (int t = 0; t < MT; ++t)
#pragma unroll
      for (int u = 0; u < 2; ++u)
        acc[t][u] = __builtin_amdgcn_mfma_scale_f32_16x16x128_f8f6f4(wf[u], xf[t], acc[t][u], 0, 0, 0, 0, 0, 0);
  };
#pragma unroll
  for (int t = 0; t < MT; ++t) acc[t][0] = acc[t][1] = v4f{0.f, 0.f, 0.f, 0.f};
  v8i xa[MT], wa[2], xb[MT], wb[2];
#pragma unroll
  for (int t = 0; t < MT; ++t) xb[t] = v8i{0, 0, 0, 0, 0, 0, 0, 0};
  wb[0] = wb[1] = v8i{0, 0, 0, 0, 0, 0, 0, 0};
  unsigned cur = 0;
#define STEP(XN, WN, XO, WO) do { \
    wait_lgkm_barrier(); \
    rd(cur, XN, WN); \
    mma(XO, WO); \
    __builtin_amdgcn_sched_group_barrier(0x002, 4, 0); \
    _Pragma("unroll") for (int i = 0; i < 2 * MT; ++i) { \
      __builtin_amdgcn_sched_group_barrier(0x008, 1, 0); \
      __builtin_amdgcn_sched_group_barrier(0x100, 2, 0); \
    } \
    __builtin_amdgcn_sched_barrier(0); \
    cur = (cur + SLOT_BYTES == SLOTS * SLOT_BYTES) ? 0u : cur + SLOT_BYTES; \
  } while (0)
#pragma unroll 1
  for (int k = 0; k < NSTEP; k += 2) {
    STEP(xa, wa, xb, wb);
    STEP(xb, wb, xa, wa);
  }
  mma(xb, wb);
}

__device__ __forceinline__ void store_tile(rsrc_t dst, int nt, unsigned so, bool remote, float s, int lane, v4f (&acc)[MT][2]) {
  const unsigned voff = so + (unsigned)(lane & 7) * 2048u + (unsigned)(lane & 8) * 8u + (unsigned)(lane >> 4) * 16u;
#pragma unroll
  for (int t = 0; t < MT; ++t) {
    if (t < nt) {
      v4u o1, o2;
#pragma unroll
      for (int r = 0; r < 4; ++r) {
        const int a = __builtin_bit_cast(int, acc[t][0][r] * s), b = __builtin_bit_cast(int, acc[t][1][r] * s);
        o1[r] = (unsigned)__builtin_amdgcn_update_dpp(a, b, 0x118, 0xf, 0xc, false);
        o2[r] = (unsigned)__builtin_amdgcn_update_dpp(b, a, 0x108, 0xf, 0x3, false);
      }
      if (remote) { if (2 * (nt - t) <= TAILN) wait_vm<1>(); else wait_vm<0>(); }
      store16(o1, dst, voff + (unsigned)t * 16u * 2048u);
      if (remote) { if (2 * (nt - t) - 1 <= TAILN) wait_vm<1>(); else wait_vm<0>(); }
      store16(o2, dst, voff + (unsigned)t * 16u * 2048u + 8u * 2048u);
    }
  }
}

template <int W>
__device__ __forceinline__ bool wait_flags(const unsigned char* myws, unsigned off, bool need, unsigned call, int lane) {
  unsigned* tmo = (unsigned*)(myws + WS_TMO);
  const rsrc_t r = RSRC(myws, WS_TMO);
  const unsigned long long t0 = now();
  for (;;) {
    bool ok;
    if (W == 1) {
      ok = (int)(__builtin_amdgcn_raw_buffer_load_b32(r, off, 0, 17) - call) >= 0;
    } else {
      const v4u v = __builtin_amdgcn_raw_buffer_load_b128(r, off, 0, 17);
      ok = (int)(v[0] - call) >= 0 && (int)(v[1] - call) >= 0 && (int)(v[2] - call) >= 0 && (int)(v[3] - call) >= 0;
    }
    const unsigned t = __hip_atomic_load(tmo, __ATOMIC_RELAXED, __HIP_MEMORY_SCOPE_AGENT);
    asm volatile("" ::: "memory");
    if (__all(!need || ok)) return true;
    if (t == call || now() - t0 > DEADLINE) {
      if (lane == 0) __hip_atomic_store(tmo, call, __ATOMIC_RELAXED, __HIP_MEMORY_SCOPE_AGENT);
      return false;
    }
    __builtin_amdgcn_s_sleep(W == 1 ? 8 : 1);
  }
}

}

__global__ void __launch_bounds__(512, 2) gemm_a2a_kernel(Params P) {
  extern __shared__ __attribute__((aligned(16))) char smem[];
  lds_char* lds = (lds_char*)smem;
  const int lane = threadIdx.x & 63;
  const int wave8 = __builtin_amdgcn_readfirstlane(threadIdx.x >> 6);
  const int wave = wave8 & 3;
  const int b = blockIdx.x, lab = b & 7, idx = b >> 3, ct = idx & 3;
  const int rg = ((idx >> 2) + NRG) & 7;
  const int rank = P.rank & 7;
  unsigned char* myws = P.ws[0];
  unsigned char* dws = P.ws[lab];
  float* myout = P.out[0];
  float* dout = P.out[lab];
  const unsigned char* px = P.x;
  const unsigned char* pw = P.w;
  const float* psx = P.sx;
  const float* psw = P.sw;
  asm volatile("" ::"s"(rank), "s"(myws), "s"(dws), "s"(myout), "s"(dout), "s"(px), "s"(pw), "s"(psx), "s"(psw));
  const bool pusher = lab != 0 && rg >= NRG;
  const int t1 = lab == 0 ? rg * 4 : (NT_REM * rg) / NRG;
  const int n1 = lab == 0 ? 4 : (NT_REM * (rg + 1)) / NRG - t1;
  const int krot = lab * ROT_L + (lab ? (rg * 32 + NRG / 2) / NRG : rg * 4);
  const bool counts = pusher || wave8 < 4 || (lab == 0 && wave8 == 4);
  unsigned* cword = (unsigned*)(myws + WS_CALL + (size_t)(b * 8 + wave8) * 128);
  unsigned call = 0;
  float sxv = 0.f, swv = 0.f;
  Loader ld;
  if (pusher) {
    const rsrc_t xs = RSRC(px + (size_t)(ML - XR) * KD, XR * KD);
    const unsigned off = (unsigned)((rg - NRG) * 4 + ct) * 1024u + (unsigned)lane * 16u;
    __attribute__((address_space(3))) unsigned* next = (__attribute__((address_space(3))) unsigned*)(lds + LDS_BYTES - 32);
    unsigned pc[PWIN];
    v4u d[PWIN];
#pragma unroll
    for (int i = 0; i < PWIN; ++i) {
      pc[i] = (unsigned)(wave8 + 8 * i);
      d[i] = __builtin_amdgcn_raw_buffer_load_b128(xs, off + pc[i] * (NPG * 4096u), 0, 0);
      if (i == 0) call = __hip_atomic_load(cword, __ATOMIC_RELAXED, __HIP_MEMORY_SCOPE_SYSTEM);
    }
    if (wave8 < 4) { sxv = psx[0]; swv = psw[0]; }
    if (threadIdx.x == 0) *next = 8u * PWIN;
    wait_lgkm_barrier();
    call = __builtin_amdgcn_readfirstlane(call) + 1u;
    asm volatile("" : "+v"(sxv), "+v"(swv));
    const rsrc_t xd = RSRC(dws + WS_XS + (size_t)(call & 1u) * XS_BYTES + (size_t)(7 - lab) * XR * KD, XR * KD);
    do {
#pragma unroll
      for (int i = 0; i < PWIN; ++i) {
        store16(d[i], xd, off + pc[i] * (NPG * 4096u));
        unsigned nx = 0;
        if (lane == 0) nx = __hip_atomic_fetch_add(next, 1u, __ATOMIC_RELAXED, __HIP_MEMORY_SCOPE_WORKGROUP);
        pc[i] = __builtin_amdgcn_readfirstlane(nx);
        d[i] = __builtin_amdgcn_raw_buffer_load_b128(xs, off + pc[i] * (NPG * 4096u), 0, 0);
      }
    } while (pc[0] < (unsigned)NPIECE);
    wait_vm<0>();
    STAMPMAX(4);
    if (lane == 0) put_word(dws + WS_XF + (size_t)rank * 256 + (size_t)(((rg - NRG) * 4 + ct) * 8 + wave8) * 4, call);
  } else {
    if (counts) call = __hip_atomic_load(cword, __ATOMIC_RELAXED, __HIP_MEMORY_SCOPE_SYSTEM);
    if (wave8 >= 4) {
      ld.init_w(lds, pw + ((rank + lab) & 7) * ML + ct * 128, krot, wave, lane);
      ld.init_x(px, ML * KD, (unsigned)t1 * 16u, n1, wave);
      if (lab == 0) {
        const unsigned long long t = now();
        while (now() - t < L0DELAY) __builtin_amdgcn_s_sleep(2);
      }
#pragma unroll
      for (int s = 0; s < PF; ++s) { ld.issue_w(s, s * SLOT_BYTES); ld.issue_x(s, s * SLOT_BYTES); }
      ld.run<false>();
      if (lab != 0) return;
    } else {
      sxv = psx[0]; swv = psw[0];
    }
    call = __builtin_amdgcn_readfirstlane(call) + 1u;
  }
  if (counts && lane == 0) put_word(cword, call);
  if (wave8 >= 4 && lab == 0 && rg == 7) return;
  const int part2 = pusher ? rg - NRG : NPG;
  const int slot2 = pusher ? lab - 1 : rg;
  const int t2 = (TPS * part2) / (NPG + 1);
  const int n2 = (TPS * (part2 + 1)) / (NPG + 1) - t2;
  __attribute__((address_space(3))) int* flag = (__attribute__((address_space(3))) int*)(lds + LDS_BYTES - 64);
  asm volatile("" : "+v"(sxv), "+v"(swv));
  float scale = sxv * swv;
  const unsigned colbyte = (unsigned)(ct * 128 + wave * 32) * 4u;

#pragma unroll 1
  for (int job = (pusher || wave8 >= 4) ? 1 : 0; job < ((lab == 0 ? rg < 7 : pusher) ? 2 : 1); ++job) {
    unsigned orow = (unsigned)(rank * ML + t1 * 16);
    int nt = n1;
    float* obase = dout;
    if (job == 1) {
      const int src = (rank + 1 + slot2) & 7;
      if (wave8 >= 4) {
        ld.init_w(lds, pw + rank * ML + ct * 128, pusher ? lab * 2 + part2 * (32 / NPG) : krot, wave, lane);
#pragma unroll
        for (int s = 0; s < PF; ++s) ld.issue_w(s, s * SLOT_BYTES);
      }
      if (wave8 == 4) {
        const bool ok = wait_flags<1>(myws, (unsigned)WS_XF + (unsigned)src * 256u + (unsigned)lane * 4u, lane < NPG * 32, call, lane);
        __builtin_amdgcn_fence(__ATOMIC_ACQUIRE, "");
        wait_vm<0>();
        if (lane == 0) { flag[0] = ok ? 1 : 0; flag[1] = (int)call; }
      }
      wait_lgkm_barrier();
      const bool ok = __builtin_amdgcn_readfirstlane(flag[0]) != 0;
      call = (unsigned)__builtin_amdgcn_readfirstlane(flag[1]);
      if (wave8 >= 4) {
        ld.init_x(myws + WS_XS + (size_t)(call & 1u) * XS_BYTES, ok ? (unsigned)XS_BYTES : 0u, (unsigned)(slot2 * XR + t2 * 16), n2, wave);
#pragma unroll
        for (int s = 0; s < PF; ++s) ld.issue_x(s, s * SLOT_BYTES);
        ld.run<true>();
        return;
      }
      if (wave8 == 0) STAMP(3);
      nt = n2;
      obase = myout;
      orow = (unsigned)(src * ML + (ML - XR) + t2 * 16);
      if (!ok) scale = __builtin_nanf("");
    }
    v4f acc[MT][2];
    mma_tile(lds, wave, lane, acc);
    if (wave == 0 && job == 0) STAMP(1);
    store_tile(RSRC(obase, NR * ML * ML * 4), nt, orow * 2048u + colbyte, job == 0 && lab != 0, scale, lane, acc);
  }
  if (lab != 0 && !pusher) {
    wait_vm<0>();
    if (wave == 0) STAMP(2);
    if (lane == 0) put_word(dws + WS_OF + (size_t)rank * 512 + (size_t)((rg * 4 + ct) * 4 + wave) * 4, call);
    return;
  }
  if (wave == 0) STAMP(2);
  if (lab == 0 && rg == 7 && ct == 0) {
    wait_vm<0>();
    const bool ok = wait_flags<4>(myws, (unsigned)WS_OF + (unsigned)wave * 1024u + (unsigned)lane * 16u,
                                  wave * 2 + (lane >> 5) != rank && (lane & 31) < NRG * 4, call, lane);
    if (!ok && lane == 0) myout[(size_t)(rank * ML + 448) * ML + wave * 32] = __builtin_nanf("");
    STAMPMAX(5);
  }
}

extern "C" void kernel_launch(int rank, int nranks, ncclComm_t comm, hipStream_t stream,
                              void* const* d_in, const int* in_sizes, int n_in,
                              void* d_out, int out_size,
                              const int* in_dtypes, int out_dtype,
                              void* const* d_out_all, void* const* d_ws_all, size_t ws_bytes) {
  (void)comm; (void)in_dtypes; (void)out_dtype; (void)d_out;
  if (nranks != NR || n_in != 4 || in_sizes[0] != ML * KD || in_sizes[1] != KD * ND || out_size != NR * ML * ML ||
      ws_bytes < WS_NEED || (unsigned)rank >= (unsigned)NR) {
    fprintf(stderr, "kernel_launch: unexpected problem shape\n");
    return;
  }
  static bool init = false;
  if (!init) {
    HIP_CHECK(hipFuncSetAttribute((const void*)gemm_a2a_kernel, hipFuncAttributeMaxDynamicSharedMemorySize, LDS_BYTES));
    init = true;
  }
  Params P;
  P.x = (const unsigned char*)d_in[0];
  P.w = (const unsigned char*)d_in[1];
  P.sx = (const float*)d_in[2];
  P.sw = (const float*)d_in[3];
  for (int p = 0; p < NR; ++p) {
    P.out[p] = (float*)d_out_all[(rank + p) % NR];
    P.ws[p] = (unsigned char*)d_ws_all[(rank + p) % NR];
  }
  P.rank = rank;
  hipLaunchKernelGGL(gemm_a2a_kernel, dim3(256), dim3(512), LDS_BYTES, stream, P);
  HIP_CHECK(hipGetLastError());
}
```
